```python
import numpy as np

kernel_name = "gpu_ag_gemm_m4096_k4096_n16384_e4m3e5m2_dqsilu"


ABSMAX_RTOL = 0.05

COMPUTE_DTYPE = "float32"
EPILOGUE = "dequant_silu"
IN_QUANT = None
OUT_QUANT = None
IN_FP8 = "e4m3e5m2"
IN_DTYPES = {"x": "e4m3", "w_mat": "e5m2",
             "scale_x": "float32", "scale_w": "float32"}
WORLD_SIZE = 8

def _snap_fp8(x, mant_bits, max_abs, min_normal):
    a = np.abs(x.astype(np.float64))
    _m, _e = np.frexp(a)
    q = np.where(a >= min_normal, np.ldexp(1.0, _e - 1 - mant_bits),
                 min_normal * 2.0 ** -mant_bits)
    snapped = np.minimum(np.round(a / q) * q, max_abs)
    return (np.sign(x) * snapped).astype(np.float32)

def setup_inputs():

    x = _snap_fp8(np.random.randn(4096, 4096).astype(np.float32), 3, 448.0, 2.0 ** -6)
    w_mat = _snap_fp8(np.random.randn(4096, 16384).astype(np.float32), 2, 57344.0, 2.0 ** -14)
    scale_x = (0.02 * (1.0 + np.random.rand(1))).astype(np.float32)
    scale_w = (0.02 * (1.0 + np.random.rand(1))).astype(np.float32)
    return {"x": x, "w_mat": w_mat, "scale_x": scale_x, "scale_w": scale_w}

def reference(x, w_mat, scale_x, scale_w):
    acc = (x @ w_mat).astype(np.float32)
    y = (acc * (scale_x * scale_w)).astype(np.float32)
    return (y / (1.0 + np.exp(-np.clip(y, -60.0, 60.0)))).astype(np.float32)
```

```cpp
#include <hip/hip_runtime.h>
#include <rccl/rccl.h>
#include <cstdio>
#include <cstdint>

#define HIP_CHECK(expr) do { hipError_t _e = (expr); if (_e != hipSuccess) { \
    fprintf(stderr, "HIP error %s at %s:%d\n", hipGetErrorString(_e), __FILE__, __LINE__); \
    return; } } while (0)

typedef unsigned u32x4 __attribute__((ext_vector_type(4)));
typedef int v2i __attribute__((ext_vector_type(2)));
typedef int v4i __attribute__((ext_vector_type(4)));
typedef int v8i __attribute__((ext_vector_type(8)));
typedef float v4f __attribute__((ext_vector_type(4)));
#define LAS __attribute__((address_space(3)))
#define GAS __attribute__((address_space(1)))
#define CAS __attribute__((address_space(4)))
#define SB() __builtin_amdgcn_sched_barrier(0)

#define OUT_AUX 17
#define X_AUX 17
#define DK 4
#define ST_KS 4
#define DBG_ALLSTEPS 0
#define DBG_X 0
#define MAPR 8

constexpr int NR = 8;
constexpr int MS = 512, KD = 4096, NC = 2048;
constexpr unsigned XB = MS * KD;
constexpr int NBLK = MS / 16;
constexpr int NPW = 16;
constexpr int GRID = 256;
constexpr size_t OFF_CTR = 0;
constexpr size_t OFF_TMO = 1024 * 128;
constexpr size_t OFF_FLAGS = OFF_TMO + 4096;
constexpr size_t OFF_DBG = 256 << 10;
constexpr size_t OFF_STAGE = 1 << 20;
constexpr size_t WS_NEED = OFF_STAGE + 2 * (size_t)NR * XB;
constexpr unsigned long long DEADLINE = 40000000ull;

struct Params {
    char* ws[NR];
    const char* x; const char* w; const float* sx; const float* sw; float* out;
    unsigned rank; unsigned nranks;
};

template <int N> __device__ __forceinline__ void waitv() { asm volatile("s_waitcnt vmcnt(%0)" :: "i"(N) : "memory"); }

__device__ __forceinline__ __amdgpu_buffer_rsrc_t mkrsrc(const void* p, unsigned bytes) {
    return __builtin_amdgcn_make_buffer_rsrc((void*)p, (short)0, (int)bytes, 0x00020000);
}

__device__ __forceinline__ unsigned ticket_issue(char* ctr) {
    unsigned t; unsigned long long sv;
    asm volatile("s_mov_b64 %1, exec\n\ts_mov_b64 exec, 1\n\tglobal_atomic_add %0, %2, %3, %4 sc0\n\ts_mov_b64 exec, %1"
                 : "=&v"(t), "=&s"(sv) : "v"(0), "v"(1), "s"(ctr) : "memory");
    return t;
}
__device__ __forceinline__ unsigned ticket_get(unsigned t) {
    asm volatile("" : "+v"(t));
    return __builtin_amdgcn_readfirstlane(t) + 1;
}

#define TS(slot) do { } while (0)

__global__ __launch_bounds__(256, 1) void ag_gemm_silu(Params p) {
    __shared__ __attribute__((aligned(16))) char lds[4 * 32768];
    const int lane = threadIdx.x & 63;
    const int wid = __builtin_amdgcn_readfirstlane(threadIdx.x >> 6);
    const int bid = blockIdx.x;
    const int gwave = bid * 4 + wid;
    char* const me = p.ws[p.rank];
    TS(0);
    const int role = bid >> 5, cs = (bid & 7) * 4 + ((bid >> 3) & 3), pidx = bid;

    if (role == 0) {
        const int gw = pidx * 4 + wid;
        if (gw >= 7 * NPW) return;
        const int pi = gw % 7, part = gw / 7;
        const unsigned peer = (p.rank + 1 + pi) % NR;
        if (peer >= p.nranks) return;
        const char* src = p.x + (lane & 15) * KD + (lane >> 4) * 16 + part * 256;
        u32x4 buf[4];
#define LOADB(i, b) asm volatile("global_load_dwordx4 %0, %1, off offset:%2" : "=v"(buf[i]) : "v"(src + (size_t)(b) * 65536), "i"((i) * 64) : "memory")
        LOADB(0, 0); LOADB(1, 0); LOADB(2, 0); LOADB(3, 0);
        const unsigned tk = ticket_issue(me + OFF_CTR + (size_t)gwave * 128);
        waitv<0>();
        const unsigned call = ticket_get(tk);
        char* const pw = p.ws[peer];
        char* dst = pw + OFF_STAGE + ((size_t)(call & 1) * NR + p.rank) * XB + part * 4096 + lane * 16;
        unsigned* flg = (unsigned*)(pw + OFF_FLAGS + (size_t)(p.rank * NPW + part) * 128);
        const unsigned base = call * 64;
        TS(1);
#define STOREB(i) asm volatile("global_store_dwordx4 %0, %1, off offset:%2 sc0 sc1\n\ts_nop 1" :: "v"(d), "v"(buf[i]), "i"((i) * 1024) : "memory")
        for (int b = 0; b < NBLK; ++b) {
            char* d = dst + (size_t)b * 65536;
            const int bn = b + 1 < NBLK ? b + 1 : b;
            waitv<5>(); asm volatile("" : "+v"(buf[0])); STOREB(0); LOADB(0, bn);
            waitv<5>(); asm volatile("" : "+v"(buf[1])); STOREB(1); LOADB(1, bn);
            waitv<5>();
            if (lane == 0) { const unsigned v = base + b; asm volatile("global_store_dword %0, %1, off sc0 sc1\n\ts_nop 1" :: "v"(flg), "v"(v) : "memory"); }
            asm volatile("" : "+v"(buf[2])); STOREB(2); LOADB(2, bn);
            waitv<5>(); asm volatile("" : "+v"(buf[3])); STOREB(3); LOADB(3, bn);
        }
        waitv<0>();
        if (lane == 0) { const unsigned v = base + NBLK; asm volatile("global_store_dword %0, %1, off sc0 sc1\n\ts_nop 1" :: "v"(flg), "v"(v) : "memory"); }
        TS(2);
        return;
    }

    const unsigned srcr = (p.rank + role) % NR;
    const int g = lane >> 4, li = lane & 15;
    char* const myl = lds + wid * 32768;
    const unsigned tk = ticket_issue(me + OFF_CTR + (size_t)gwave * 128);
    const float scale = *(const CAS float*)p.sx * *(const CAS float*)p.sw;

    const int nown = role <= 2 ? 6 : 4;
    const int own0 = role <= 2 ? (role - 1) * 6 : 12 + (role - 3) * 4;
    const int nb = nown + NBLK;
    const __amdgpu_buffer_rsrc_t rs_own = mkrsrc(p.x, XB);
    const __amdgpu_buffer_rsrc_t rs_nul = mkrsrc(p.x, 0);
    const __amdgpu_buffer_rsrc_t rs_out = mkrsrc(p.out, (unsigned)(NR * MS * NC * 4));
    const int vo_own = li * KD + g * 16 + wid * 1024;
    const int vo_rem = lane * 16 + wid * 16384;
    u32x4 xb[2][16];

    v8i wf[8][4];
    {
        const int kr = lane >> 2;
        const __amdgpu_buffer_rsrc_t rs_w = mkrsrc(p.w, (unsigned)(KD * NC));
        const int wvA = (wid * 1024 + kr) * NC + cs * 64 + (((lane & 3) ^ ((kr >> 2) & 1)) * 16);
        const int wvB = wvA ^ 32;
        const unsigned lbase = (unsigned)(size_t)(LAS char*)myl;
#define WLOAD(r) do { _Pragma("unroll") for (int j = 0; j < 16; ++j) \
            asm volatile("s_mov_b32 m0, %0\n\ts_nop 0\n\tbuffer_load_dwordx4 %1, %2, %3 offen lds" \
                :: "s"(lbase + ((r) & 1) * 16384 + j * 1024), "v"((j & 1) ? wvB : wvA), "s"(rs_w), "s"(((r) * 256 + j * 16) * NC) : "memory"); } while (0)
        const int q = li >> 1, h = li & 1;
        const int f0 = ((q >> 2) & 1) | ((g & 1) << 1);
        const int rbase = (16 * g + q) * 64 + 8 * h;
#define WREAD(r) do { _Pragma("unroll") for (int ks = 0; ks < 2; ++ks) _Pragma("unroll") for (int t = 0; t < 4; ++t) { \
            v2i e[4]; \
            _Pragma("unroll") for (int u = 0; u < 4; ++u) \
                e[u] = __builtin_amdgcn_ds_read_tr8_b64_v2i32((LAS v2i*)(myl + ((r) & 1) * 16384 + ks * 8192 + (u >> 1) * 4096 + (u & 1) * 512 + rbase + ((t ^ f0) * 16))); \
            wf[(r) * 2 + ks][t] = (v8i){e[0][0], e[0][1], e[1][0], e[1][1], e[2][0], e[2][1], e[3][0], e[3][1]}; } \
            asm volatile("s_waitcnt lgkmcnt(0)" ::: "memory"); } while (0)
        WLOAD(0); WLOAD(1);
        waitv<16>(); WREAD(0); WLOAD(2);
        waitv<16>(); WREAD(1); WLOAD(3);
#pragma unroll
        for (int q2 = 0; q2 < 2 * DK; ++q2) { xb[0][q2] = __builtin_amdgcn_raw_buffer_load_b128(rs_own, vo_own, own0 * 65536 + q2 * 64, X_AUX); SB(); }
        waitv<16 + 2 * DK>(); WREAD(2);
        waitv<2 * DK>(); WREAD(3);
    }
    TS(1);
    const unsigned call = ticket_get(tk);
    const __amdgpu_buffer_rsrc_t rs_rem = mkrsrc(me + OFF_STAGE + ((size_t)(call & 1) * NR + srcr) * XB, XB);
    unsigned* const tmo = (unsigned*)(me + OFF_TMO);
    const unsigned* const pollp = lane < 4 ? (const unsigned*)(me + OFF_FLAGS + (size_t)(srcr * NPW + wid * 4 + lane) * 128) : tmo;
    const unsigned base = call * 64;
    int ready = nown;
    v4f cin = {0.f, 0.f, 0.f, 0.f};
    const int wr_off = g * 256 + ((li ^ g) * 16);
    const int mrow = 4 * wid + g;
    const int rd_off = (li >> 2) * 1024 + (li & 3) * 256 + ((mrow ^ li) * 16);
    const int vo_out = mrow * (NC * 4) + cs * 256 + li * 16;

    v4f accA[4], accB[4], part[4], res = {0.f, 0.f, 0.f, 0.f};
#pragma unroll
    for (int t = 0; t < 4; ++t) accB[t] = (v4f){0.f, 0.f, 0.f, 0.f};

#define REMBLK(j) (j)
#define WAIT_READY(i) do { if (ready <= (i)) { \
        const unsigned long long t0 = __builtin_amdgcn_s_memrealtime(); \
        while (true) { \
            const unsigned v = __hip_atomic_load(pollp, __ATOMIC_RELAXED, __HIP_MEMORY_SCOPE_SYSTEM); \
            const unsigned f0_ = __builtin_amdgcn_readlane(v, 0), f1_ = __builtin_amdgcn_readlane(v, 1), f2_ = __builtin_amdgcn_readlane(v, 2), f3_ = __builtin_amdgcn_readlane(v, 3); \
            const unsigned fm = min(min(f0_, f1_), min(f2_, f3_)); \
            if (fm > base) ready = nown + (int)min(fm - base, (unsigned)NBLK); \
            if (ready > (i)) break; \
            if ((unsigned)__builtin_amdgcn_readlane(v, 4) == call || __builtin_amdgcn_s_memrealtime() - t0 > DEADLINE) { \
                if (lane == 0) __hip_atomic_store(tmo, call, __ATOMIC_RELAXED, __HIP_MEMORY_SCOPE_AGENT); \
                ready = nb; cin = (v4f){__builtin_nanf(""), __builtin_nanf(""), __builtin_nanf(""), __builtin_nanf("")}; \
                asm volatile("s_nop 3" : "+v"(cin)); break; } \
            __builtin_amdgcn_s_sleep(1); \
        } } \
        asm volatile("" ::: "memory"); } while (0)
#define LD_SETUP(i) do { \
        const bool own_ = (i) < nown; \
        rs_ = own_ ? rs_own : (i) < nb ? rs_rem : rs_nul; \
        vo_ = own_ ? vo_own : vo_rem; \
        so_ = (own_ ? own0 + (i) : REMBLK((i) - nown)) * 65536; qs_ = own_ ? 64 : 1024; } while (0)
#define LD(bf, q_) xb[bf][q_] = __builtin_amdgcn_raw_buffer_load_b128(rs_, vo_, so_ + (q_) * qs_, X_AUX)
#define RED_WRITE(acc, bf) do { asm volatile("s_nop 3" : "+v"(acc[0]), "+v"(acc[1]), "+v"(acc[2]), "+v"(acc[3])); _Pragma("unroll") for (int t = 0; t < 4; ++t) \
        *(LAS v4f*)((LAS char*)myl + (bf) * 4096 + t * 1024 + (wr_off ^ (t * 64))) = acc[t]; } while (0)
#define BARRIER "s_waitcnt lgkmcnt(0)\n\ts_barrier"
#define RED_READ(bf) do { asm volatile(BARRIER ::: "memory"); \
        _Pragma("unroll") for (int w = 0; w < 4; ++w) part[w] = *(LAS v4f*)((LAS char*)lds + w * 32768 + (bf) * 4096 + rd_off); } while (0)
#define RED_SUM() do { res = (part[0] + part[1]) + (part[2] + part[3]); asm volatile("" : "+v"(res)); } while (0)
#define EPI(r) do { \
        const float y = res[r] * scale; \
        const float e = __builtin_amdgcn_exp2f(__builtin_amdgcn_fmed3f(y, -60.f, 60.f) * -1.44269504088896f); \
        res[r] = y * __builtin_amdgcn_rcpf(1.f + e); asm volatile("" : "+v"(res[r])); } while (0)
#define STORE_RES(i) do { \
        const int row0_ = (i) < nown ? (int)p.rank * MS + (own0 + (i)) * 16 : (int)srcr * MS + ((i) - nown) * 16; \
        __builtin_amdgcn_raw_buffer_store_b128(__builtin_bit_cast(u32x4, res), (i) >= 0 ? rs_out : rs_nul, vo_out, row0_ * (NC * 4), OUT_AUX); } while (0)
#define MF(acc, t, ks) do { if (ks == 0) asm volatile("v_mfma_f32_16x16x128_f8f6f4 %0, %1, %2, %3 cbsz:1" : "=&v"(acc[t]) : "a"(wf[ks][t]), "v"(xv), "v"(cin)); \
        else asm volatile("v_mfma_f32_16x16x128_f8f6f4 %0, %1, %2, %0 cbsz:1" : "+v"(acc[t]) : "a"(wf[ks][t]), "v"(xv)); } while (0)
#define STEP(i, bf, acc, accp) do { \
        _Pragma("unroll") for (int ks = 0; ks < 8; ++ks) { \
            if (ks == 8 - DK) { if ((i) + 1 < nb) WAIT_READY((i) + 1); LD_SETUP((i) + 1); SB(); } \
            const int lb_ = ks + DK >= 8 ? (bf) ^ 1 : (bf), lq_ = 2 * ((ks + DK) & 7); \
            const u32x4 lo_ = xb[bf][2 * ks], hi_ = xb[bf][2 * ks + 1]; \
            const v8i xv = {(int)lo_[0], (int)lo_[1], (int)lo_[2], (int)lo_[3], (int)hi_[0], (int)hi_[1], (int)hi_[2], (int)hi_[3]}; \
            LD(lb_, lq_); MF(acc, 0, ks); MF(acc, 1, ks); SB(); \
            if (ks == 1 && !(DBG_X & 1)) RED_WRITE(accp, (bf) ^ 1); \
            if (ks == 3 && !(DBG_X & 2)) { EPI(0); EPI(1); } \
            if (ks == ST_KS && !(DBG_X & 4)) STORE_RES((i) - 1); \
            SB(); \
            LD(lb_, lq_ + 1); MF(acc, 2, ks); MF(acc, 3, ks); SB(); \
            if (ks == 1 && !(DBG_X & 1)) RED_READ((bf) ^ 1); \
            if (ks == 2 && !(DBG_X & 1)) RED_SUM(); \
            if (ks == 3 && !(DBG_X & 2)) { EPI(2); EPI(3); } \
            SB(); } \
        if (DBG_ALLSTEPS || (i) == nown - 1) TS(DBG_ALLSTEPS ? 2 + (i) : 2); \
        } while (0)

    __amdgpu_buffer_rsrc_t rs_; int vo_, so_, qs_;
    LD_SETUP(0);
    for (int i = 0; i < nb; i += 2) { STEP(i, 0, accA, accB); STEP(i + 1, 1, accB, accA); }
    asm volatile("s_nop 15");
    RED_WRITE(accB, 1); RED_READ(1);
    RED_SUM(); EPI(0); EPI(1); EPI(2); EPI(3);
    STORE_RES(nb - 1);
    SB();
#pragma unroll
    for (int q = 0; q < 2 * DK; ++q) asm volatile("" :: "v"(xb[0][q]));
}

extern "C" void kernel_launch(int rank, int nranks, ncclComm_t comm, hipStream_t stream,
                              void* const* d_in, const int* in_sizes, int n_in,
                              void* d_out, int out_size, const int* in_dtypes, int out_dtype,
                              void* const* d_out_all, void* const* d_ws_all, size_t ws_bytes) {
    (void)comm; (void)in_dtypes; (void)out_dtype; (void)d_out_all;
    if (nranks != NR || rank < 0 || rank >= NR || n_in < 4 || ws_bytes < WS_NEED || in_sizes[0] != (int)XB || in_sizes[1] != KD * NC
        || out_size != NR * MS * NC) { fprintf(stderr, "unexpected problem shape\n"); return; }
    Params p;
    for (int r = 0; r < NR; ++r) p.ws[r] = (char*)d_ws_all[r];
    p.x = (const char*)d_in[0]; p.w = (const char*)d_in[1]; p.sx = (const float*)d_in[2]; p.sw = (const float*)d_in[3];
    p.out = (float*)d_out; p.rank = (unsigned)rank; p.nranks = (unsigned)nranks;
    ag_gemm_silu<<<GRID, 256, 0, stream>>>(p);
    HIP_CHECK(hipGetLastError());
}
```

```cpp
#include <hip/hip_runtime.h>
#include <rccl/rccl.h>
#include <cstdio>
#include <cstdint>

#define HIP_CHECK(expr) do { hipError_t _e = (expr); if (_e != hipSuccess) { \
    fprintf(stderr, "HIP error %s at %s:%d\n", hipGetErrorString(_e), __FILE__, __LINE__); \
    return; } } while (0)

typedef unsigned u32x4 __attribute__((ext_vector_type(4)));
typedef int v2i __attribute__((ext_vector_type(2)));
typedef int v4i __attribute__((ext_vector_type(4)));
typedef int v8i __attribute__((ext_vector_type(8)));
typedef float v4f __attribute__((ext_vector_type(4)));
#define LAS __attribute__((address_space(3)))
#define GAS __attribute__((address_space(1)))
#define CAS __attribute__((address_space(4)))
#define SB() __builtin_amdgcn_sched_barrier(0)

#define OUT_AUX 17
#define X_BITS "sc0 sc1"
#define ROT_END 24
#define WPF 1
#define SLEEP 1
#define RS0 10
#define DBG_ALLSTEPS 0
#define DBG_X 0
#define MAPR 8

constexpr int NR = 8;
constexpr int MS = 512, KD = 4096, NC = 2048;
constexpr unsigned XB = MS * KD;
constexpr int NBLK = MS / 16;
constexpr int NPW = 16;
constexpr int GRID = 256;
constexpr int LW = 40960;
constexpr size_t OFF_CTR = 0;
constexpr size_t OFF_TMO = 1024 * 128;
constexpr size_t OFF_FLAGS = OFF_TMO + 4096;
constexpr size_t OFF_DBG = 256 << 10;
constexpr size_t OFF_STAGE = 1 << 20;
constexpr size_t WS_NEED = OFF_STAGE + 2 * (size_t)NR * XB;
constexpr unsigned long long DEADLINE = 40000000ull;

struct Params {
    char* dst[NR];
    const char* x; const char* w; const float* sx; const float* sw; float* out;
    unsigned rank; unsigned pad;
};

template <int N> __device__ __forceinline__ void waitv() { asm volatile("s_waitcnt vmcnt(%0)" :: "i"(N) : "memory"); }

__device__ __forceinline__ __amdgpu_buffer_rsrc_t mkrsrc(const void* p, unsigned bytes) {
    return __builtin_amdgcn_make_buffer_rsrc((void*)p, (short)0, (int)bytes, 0x00020000);
}

__device__ __forceinline__ unsigned ticket_issue(char* ctr) {
    unsigned t; unsigned long long sv;
    asm volatile("s_mov_b64 %1, exec\n\ts_mov_b64 exec, 1\n\tglobal_atomic_add %0, %2, %3, %4 sc0\n\ts_mov_b64 exec, %1"
                 : "=&v"(t), "=&s"(sv) : "v"(0), "v"(1), "s"(ctr) : "memory");
    return t;
}
__device__ __forceinline__ unsigned ticket_get(unsigned t) {
    asm volatile("" : "+v"(t));
    return __builtin_amdgcn_readfirstlane(t) + 1;
}

#define TS(slot) do { } while (0)

__global__ __launch_bounds__(256, 1) void ag_gemm_silu(Params p) {
    __shared__ __attribute__((aligned(16))) char lds[4 * LW];
    const int lane = threadIdx.x & 63;
    const int wid = __builtin_amdgcn_readfirstlane(threadIdx.x >> 6);
    const int bid = blockIdx.x;
    const int gwave = bid * 4 + wid;
    char* const me = p.dst[NR - 1];
    TS(0);
    const int role = bid >> 5, cs = (bid & 7) * 4 + ((bid >> 3) & 3), pidx = bid;

    if (role == 0) {
        const int gw = pidx * 4 + wid;
        if (gw >= 7 * NPW) return;
        const int pi = gw % 7, part = gw / 7;
        const char* src = p.x + (lane & 15) * KD + (lane >> 4) * 16 + part * 256;
        u32x4 buf[4];
#define LOADB(i, b) asm volatile("global_load_dwordx4 %0, %1, off offset:%2" : "=v"(buf[i]) : "v"(src + (size_t)(b) * 65536), "i"((i) * 64) : "memory")
        LOADB(0, 0); LOADB(1, 0); LOADB(2, 0); LOADB(3, 0);
        const unsigned tk = ticket_issue(me + OFF_CTR + (size_t)gwave * 128);
        waitv<0>();
        const unsigned call = ticket_get(tk);
        char* const pw = p.dst[pi];
        char* dst = pw + OFF_STAGE + ((size_t)(call & 1) * NR + p.rank) * XB + part * 4096 + lane * 16;
        unsigned* flg = (unsigned*)(pw + OFF_FLAGS + (size_t)(p.rank * NPW + part) * 128);
        const unsigned base = call * 64;
        TS(1);
#define STOREB(i) asm volatile("global_store_dwordx4 %0, %1, off offset:%2 sc0 sc1\n\ts_nop 1" :: "v"(d), "v"(buf[i]), "i"((i) * 1024) : "memory")
        for (int b = 0; b < NBLK; ++b) {
            char* d = dst + (size_t)b * 65536;
            const int bn = b + 1 < NBLK ? b + 1 : b;
            waitv<5>(); asm volatile("" : "+v"(buf[0])); STOREB(0); LOADB(0, bn);
            waitv<5>(); asm volatile("" : "+v"(buf[1])); STOREB(1); LOADB(1, bn);
            waitv<5>();
            if (lane == 0) { const unsigned v = base + b; asm volatile("global_store_dword %0, %1, off sc0 sc1\n\ts_nop 1" :: "v"(flg), "v"(v) : "memory"); }
            asm volatile("" : "+v"(buf[2])); STOREB(2); LOADB(2, bn);
            waitv<5>(); asm volatile("" : "+v"(buf[3])); STOREB(3); LOADB(3, bn);
        }
        waitv<0>();
        if (lane == 0) { const unsigned v = base + NBLK; asm volatile("global_store_dword %0, %1, off sc0 sc1\n\ts_nop 1" :: "v"(flg), "v"(v) : "memory"); }
        TS(2);
        return;
    }

    const unsigned srcr = (p.rank + role) % NR;
    const int g = lane >> 4, li = lane & 15, sub = cs & 3;
    char* const myl = lds + wid * LW;
    const unsigned lbase = (unsigned)(size_t)(LAS char*)myl;
    const unsigned tk = ticket_issue(me + OFF_CTR + (size_t)gwave * 128);
    const __amdgpu_buffer_rsrc_t rs_w = mkrsrc(p.w, (unsigned)(KD * NC));
    unsigned pf0 = 0, pf1 = 0;
    if (WPF) {
        const int v0 = (((role - 1) * 16 + sub * 4 + wid) * 32 + (lane >> 1)) * NC + (cs >> 2) * 256 + (lane & 1) * 128;
        asm volatile("buffer_load_dword %0, %1, %2, 0 offen" : "=v"(pf0) : "v"(v0), "s"(rs_w) : "memory");
        asm volatile("buffer_load_dword %0, %1, %2, 0 offen" : "=v"(pf1) : "v"(v0 + 3584 * NC), "s"(rs_w) : "memory");
    }
    u32x4 xb[2][16];
    v8i wf[8][4];
    const bool dummy = role >= 5;
#define OWN_SO(j) (((j) == 0 ? (dummy ? NBLK - 1 : 27 + role) : 7 * (((j) - 1 + sub) & 3) + role - 1) * 65536)
    int vrow[16];
#pragma unroll
    for (int r = 0; r < 16; ++r) vrow[r] = r * KD + wid * 1024 + ((lane ^ r) * 16);
    __amdgpu_buffer_rsrc_t rs_own = mkrsrc(p.x, XB);
    int so_d;
#define DMA(bf, r) asm volatile("s_add_u32 m0, %0, %1\n\ts_nop 0\n\tbuffer_load_dwordx4 %2, %3, %4 offen lds" \
        :: "s"(lbase), "i"(8192 + (bf) * 16384 + (r) * 1024), "v"(vrow[r]), "s"(rs_own), "s"(so_d) : "memory", "scc")
    int xa[4];
#pragma unroll
    for (int j = 0; j < 4; ++j) xa[j] = li * 1024 + (((4 * j + g) ^ li) * 16);
#define XRD(bf, q_) xb[bf][q_] = *(LAS u32x4*)((LAS char*)myl + 8192 + (bf) * 16384 + ((q_) >> 2) * 256 + xa[(q_) & 3])

    {
        const int kr = lane >> 2;
        const int wvA = (wid * 1024 + kr) * NC + cs * 64 + (((lane & 3) ^ ((kr >> 2) & 1)) * 16);
        const int wvB = wvA ^ 32;
#define WLOAD(r) do { _Pragma("unroll") for (int j = 0; j < 16; ++j) \
            asm volatile("s_add_u32 m0, %0, %1\n\ts_nop 0\n\tbuffer_load_dwordx4 %2, %3, %4 offen lds" \
                :: "s"(lbase), "i"(8192 + ((r) & 1) * 16384 + j * 1024), "v"((j & 1) ? wvB : wvA), "s"(rs_w), "s"(((r) * 256 + j * 16) * NC) : "memory", "scc"); } while (0)
        const int q = li >> 1, h = li & 1;
        const int f0 = ((q >> 2) & 1) | ((g & 1) << 1);
        const int rbase = (16 * g + q) * 64 + 8 * h;
#define WREAD(r) do { _Pragma("unroll") for (int ks = 0; ks < 2; ++ks) _Pragma("unroll") for (int t = 0; t < 4; ++t) { \
            v2i e[4]; \
            _Pragma("unroll") for (int u = 0; u < 4; ++u) \
                e[u] = __builtin_amdgcn_ds_read_tr8_b64_v2i32((LAS v2i*)(myl + 8192 + ((r) & 1) * 16384 + ks * 8192 + (u >> 1) * 4096 + (u & 1) * 512 + rbase + ((t ^ f0) * 16))); \
            wf[(r) * 2 + ks][t] = (v8i){e[0][0], e[0][1], e[1][0], e[1][1], e[2][0], e[2][1], e[3][0], e[3][1]}; } \
            asm volatile("s_waitcnt lgkmcnt(0)" ::: "memory"); } while (0)
        WLOAD(0); WLOAD(1);
        waitv<16>(); asm volatile("" :: "v"(pf0), "v"(pf1)); WREAD(0); WLOAD(2);
        waitv<16>(); WREAD(1); WLOAD(3);
        waitv<16>(); WREAD(2);
        so_d = OWN_SO(0);
#pragma unroll
        for (int r = 0; r < 16; ++r) DMA(0, r);
        waitv<16>(); WREAD(3);
        so_d = OWN_SO(1);
#pragma unroll
        for (int r = 0; r < 16; ++r) DMA(1, r);
        waitv<16>();
#pragma unroll
        for (int q2 = 0; q2 < 16; ++q2) XRD(0, q2);
    }
    TS(1);
    const float scale = *(const CAS float*)p.sx * *(const CAS float*)p.sw;
    const unsigned call = ticket_get(tk), base = call * 64;
    const char* const stg = me + OFF_STAGE + ((size_t)(call & 1) * NR + srcr) * XB;
    const __amdgpu_buffer_rsrc_t rs_out = mkrsrc(p.out, (unsigned)(NR * MS * NC * 4));
    const int vo_x = lane * 16 + wid * 16384;
    unsigned* const tmo = (unsigned*)(me + OFF_TMO);
    const unsigned* const pollp = lane < 4 ? (const unsigned*)(me + OFF_FLAGS + (size_t)(srcr * NPW + wid * 4 + lane) * 128) : tmo;
    unsigned pv = 0;
    v4f cin = {0.f, 0.f, 0.f, 0.f};
    const int wr_off = g * 256 + ((li ^ g) * 16);
    const int mrow = 4 * wid + g;
    const int rd_off = (li >> 2) * 1024 + (li & 3) * 256 + ((mrow ^ li) * 16);
    const int vo_out = mrow * (NC * 4) + cs * 256 + li * 16;
    int vo_st = (int)0x80000000u, vo_nx = dummy ? (int)0x80000000u : vo_out;
    v4f accA[4], accB[4], part[4], res = {0.f, 0.f, 0.f, 0.f}, ex;
#pragma unroll
    for (int t = 0; t < 4; ++t) accB[t] = (v4f){0.f, 0.f, 0.f, 0.f};

    const int oo_own = (int)p.rank * (int)XB * 2, oo_rem = (int)srcr * (int)XB * 2;
    int oo_p = 0, oo_c = oo_own + 2 * OWN_SO(0), oo_n = oo_own + 2 * OWN_SO(1);
    __amdgpu_buffer_rsrc_t rs_;
    int so_q, so_2, so_3, nrec_2, nrec_3; unsigned need_2, need_3;
    unsigned known = 0;
#define QMIN(m) do { m = min(m, (unsigned)__builtin_amdgcn_mov_dpp((int)m, 0xB1, 0xF, 0xF, true)); m = min(m, (unsigned)__builtin_amdgcn_mov_dpp((int)m, 0x4E, 0xF, 0xF, true)); } while (0)
#define PICK1(n) do { int n_ = (n); asm volatile("" : "+s"(n_)); \
        so_3 = n_ < ROT_END ? (n_ & ~3) | ((n_ + sub) & 3) : n_; asm volatile("" : "+s"(so_3)); } while (0)
#define PICK2(n) do { int n_ = (n); asm volatile("" : "+s"(n_), "+s"(so_3)); \
        need_3 = n_ < NBLK ? base + 1 + so_3 : 0u; nrec_3 = n_ < NBLK ? XB : 0; so_3 *= 65536; \
        asm volatile("" : "+s"(so_3), "+s"(need_3), "+s"(nrec_3)); } while (0)
#define PICK(n) do { PICK1(n); PICK2(n); } while (0)
#define POLL() asm volatile("global_load_dword %0, %1, off sc0 sc1" : "+v"(pv) : "v"(pollp) : "memory")
#define CHECK(N) do { if (known < need_2) { \
        asm volatile("s_waitcnt vmcnt(%1)" : "+v"(pv) : "i"(N) : "memory"); \
        unsigned pm = pv; QMIN(pm); known = __builtin_amdgcn_readfirstlane(pm); \
        if (__builtin_expect(known < need_2, 0)) { \
            const unsigned long long t0 = __builtin_amdgcn_s_memrealtime(); \
            while (true) { \
                const unsigned v = __hip_atomic_load(pollp, __ATOMIC_RELAXED, __HIP_MEMORY_SCOPE_SYSTEM); \
                pm = v; QMIN(pm); known = __builtin_amdgcn_readfirstlane(pm); \
                if (known >= need_2) break; \
                if ((unsigned)__builtin_amdgcn_readlane(v, 4) == call || __builtin_amdgcn_s_memrealtime() - t0 > DEADLINE) { \
                    __hip_atomic_store(tmo, call, __ATOMIC_RELAXED, __HIP_MEMORY_SCOPE_AGENT); \
                    known = ~0u; cin = (v4f){__builtin_nanf(""), __builtin_nanf(""), __builtin_nanf(""), __builtin_nanf("")}; \
                    asm volatile("s_nop 3" : "+v"(cin)); break; } \
                __builtin_amdgcn_s_sleep(SLEEP); \
            } } } \
        asm volatile("" ::: "memory"); } while (0)
#define SHIFT() do { so_2 = so_3; need_2 = need_3; nrec_2 = nrec_3; } while (0)
#define LD(bf, q_) do { asm volatile("buffer_load_dwordx4 %0, %1, %2, %3 offen offset:%4 " X_BITS : "=v"(xb[bf][q_]) : "v"(vo_x), "s"(rs_), "s"(so_q), "i"(((q_) & 3) * 1024) : "memory"); \
        if (((q_) & 3) == 3 && (q_) < 15) { asm volatile("s_add_u32 %0, %0, 0x1000" : "+s"(so_q) :: "scc"); } } while (0)
#define RED_WRITE(acc, bf, t) *(LAS v4f*)((LAS char*)myl + (bf) * 4096 + (t) * 1024 + (wr_off ^ ((t) * 64))) = acc[t]
#define RED_BAR() asm volatile("s_waitcnt lgkmcnt(0)\n\ts_barrier" ::: "memory")
#define RED_READ(bf, w) part[w] = *(LAS v4f*)((LAS char*)lds + (w) * LW + (bf) * 4096 + rd_off)
#define ADD4(a, b) do { _Pragma("unroll") for (int r_ = 0; r_ < 4; ++r_) { a[r_] += b[r_]; asm volatile("" : "+v"(a[r_])); } } while (0)
#define EPI_A(r) do { res[r] = part[0][r] * scale; \
        ex[r] = __builtin_amdgcn_exp2f(__builtin_amdgcn_fmed3f(res[r], -60.f, 60.f) * -1.44269504088896f); asm volatile("" : "+v"(res[r]), "+v"(ex[r])); } while (0)
#define EPI_B(r) do { res[r] *= __builtin_amdgcn_rcpf(1.f + ex[r]); asm volatile("" : "+v"(res[r])); } while (0)
#define STORE_RES(vo, so) __builtin_amdgcn_raw_buffer_store_b128(__builtin_bit_cast(u32x4, res), rs_out, vo, so, OUT_AUX)
#define MF(acc, t, ks) do { if (ks == 0) asm volatile("v_mfma_f32_16x16x128_f8f6f4 %0, %1, %2, %3 cbsz:1" : "=&v"(acc[t]) : "a"(wf[ks][t]), "v"(xv), "v"(cin)); \
        else asm volatile("v_mfma_f32_16x16x128_f8f6f4 %0, %1, %2, %0 cbsz:1" : "+v"(acc[t]) : "a"(wf[ks][t]), "v"(xv)); } while (0)
#define FILL(s, bf, accp, vo) do { \
        if (!(DBG_X & 1)) { \
        if (s == 1 || s == 3 || s == 5 || s == 7) RED_WRITE(accp, (bf) ^ 1, s >> 1); \
        if (s == RS0) RED_BAR(); \
        if (s == RS0 || s == RS0 + 1) { RED_READ((bf) ^ 1, 2 * (s - RS0)); RED_READ((bf) ^ 1, 2 * (s - RS0) + 1); } \
        if (s == RS0 + 4) ADD4(part[0], part[1]); \
        if (s == RS0 + 5) ADD4(part[2], part[3]); \
        if (s == RS0 + 6) ADD4(part[0], part[2]); } \
        if (!(DBG_X & 2)) { \
        if (s >= RS0 + 7 && s < RS0 + 11) EPI_A(s - RS0 - 7); \
        if (s >= RS0 + 11 && s < RS0 + 15) EPI_B(s - RS0 - 11); } \
        if (s == RS0 + 16 && !(DBG_X & 4)) STORE_RES(vo, oo_p); \
        } while (0)
#define XV(bf, ks) const u32x4 lo_ = xb[bf][2 * ks], hi_ = xb[bf][2 * ks + 1]; \
        const v8i xv = {(int)lo_[0], (int)lo_[1], (int)lo_[2], (int)lo_[3], (int)hi_[0], (int)hi_[1], (int)hi_[2], (int)hi_[3]}
#define OSTEP(j, bf, acc, accp) do { \
        so_d = OWN_SO((j) + 2); if ((bf) == 1) rs_own = mkrsrc(p.x, (j) < 3 ? XB : 0); \
        asm volatile("s_waitcnt lgkmcnt(0)" ::: "memory"); SB(); \
        _Pragma("unroll") for (int ks = 0; ks < 8; ++ks) { XV(bf, ks); \
            _Pragma("unroll") for (int t = 0; t < 4; ++t) { const int sl = 4 * ks + t; \
                MF(acc, t, ks); SB(); \
                if (sl == 14) waitv<15 + (bf)>(); \
                if (sl < 16) DMA(bf, sl); \
                if (sl >= 14 && sl < 30) XRD((bf) ^ 1, sl - 14); \
                FILL(sl, bf, accp, vo_st); \
                if (sl == 30) { vo_st = vo_nx; vo_nx = vo_out; asm volatile("" : "+v"(vo_st), "+v"(vo_nx)); } \
                if (sl == 31 && (bf) == 0) POLL(); \
                SB(); } } \
        oo_p = oo_c; oo_c = oo_n; oo_n = oo_own + 2 * so_d; asm volatile("" : "+s"(oo_p), "+s"(oo_c), "+s"(oo_n)); \
        if (DBG_ALLSTEPS) TS(2 + (j)); \
        } while (0)
#define GSTEP(n, bf, acc, accp) do { \
        so_q = so_2; rs_ = mkrsrc(stg, nrec_2); const int so_x = so_2; SHIFT(); SB(); \
        _Pragma("unroll") for (int ks = 0; ks < 8; ++ks) { \
            LD((bf) ^ 1, 2 * ks); \
              \
            asm volatile("s_waitcnt vmcnt(%2)" : "+v"(xb[bf][2 * ks]), "+v"(xb[bf][2 * ks + 1]) : "i"((DBG_X & 4) ? 15 : 16) : "memory"); \
            XV(bf, ks); \
            _Pragma("unroll") for (int t = 0; t < 4; ++t) { const int sl = 4 * ks + t; \
                if (t == 2) LD((bf) ^ 1, 2 * ks + 1); \
                MF(acc, t, ks); SB(); \
                FILL(sl, bf, accp, vo_out); \
                if (sl == 8) { oo_n = so_x; asm volatile("" : "+s"(oo_n)); oo_n = oo_rem + 2 * oo_n; asm volatile("" : "+s"(oo_n)); } \
                if (sl == 9) PICK1((n) + 2); \
                if (sl == 13) PICK2((n) + 2); \
                if (sl == 28) CHECK((DBG_X & 4) ? 15 : 16); \
                if (sl == 30 && known < need_3) POLL(); \
                SB(); } } \
        oo_p = oo_c; oo_c = oo_n; asm volatile("" : "+s"(oo_p), "+s"(oo_c)); \
        if (DBG_ALLSTEPS) TS(6 + (n)); \
        } while (0)

    POLL();
    for (int j = 0; j < 4; j += 2) { OSTEP(j, 0, accA, accB); OSTEP(j + 1, 1, accB, accA); }
    if (!DBG_ALLSTEPS) TS(2);
    PICK(0); SHIFT(); CHECK(16); PICK(1); POLL();
    for (int n = -1; n < NBLK; n += 2) { if (n >= 0) GSTEP(n, 1, accB, accA); GSTEP(n + 1, 0, accA, accB); }
    if (!DBG_ALLSTEPS) TS(39);
    asm volatile("s_nop 15" : "+v"(accA[0]), "+v"(accA[1]), "+v"(accA[2]), "+v"(accA[3]));
    RED_WRITE(accA, 0, 0); RED_WRITE(accA, 0, 1); RED_WRITE(accA, 0, 2); RED_WRITE(accA, 0, 3);
    RED_BAR(); RED_READ(0, 0); RED_READ(0, 1); RED_READ(0, 2); RED_READ(0, 3);
    ADD4(part[0], part[1]); ADD4(part[2], part[3]); ADD4(part[0], part[2]);
    EPI_A(0); EPI_A(1); EPI_A(2); EPI_A(3); EPI_B(0); EPI_B(1); EPI_B(2); EPI_B(3);
    STORE_RES(vo_out, oo_p);
    SB();
    asm volatile("s_waitcnt vmcnt(0)" : "+v"(pv) :: "memory");
#pragma unroll
    for (int q = 0; q < 16; ++q) asm volatile("" :: "v"(xb[1][q]));
}

extern "C" void kernel_launch(int rank, int nranks, ncclComm_t comm, hipStream_t stream,
                              void* const* d_in, const int* in_sizes, int n_in,
                              void* d_out, int out_size, const int* in_dtypes, int out_dtype,
                              void* const* d_out_all, void* const* d_ws_all, size_t ws_bytes) {
    (void)comm; (void)in_dtypes; (void)out_dtype; (void)d_out_all;
    if (nranks != NR || rank < 0 || rank >= NR || n_in < 4 || ws_bytes < WS_NEED || in_sizes[0] != (int)XB || in_sizes[1] != KD * NC
        || out_size != NR * MS * NC) { fprintf(stderr, "unexpected problem shape\n"); return; }
    Params p;
    for (int i = 0; i < NR; ++i) p.dst[i] = (char*)d_ws_all[(rank + 1 + i) % NR];
    p.x = (const char*)d_in[0]; p.w = (const char*)d_in[1]; p.sx = (const float*)d_in[2]; p.sw = (const float*)d_in[3];
    p.out = (float*)d_out; p.rank = (unsigned)rank; p.pad = 0;
    ag_gemm_silu<<<GRID, 256, 0, stream>>>(p);
    HIP_CHECK(hipGetLastError());
}
```

```cpp
#include <hip/hip_runtime.h>
#include <rccl/rccl.h>
#include <cstdio>
#include <cstdint>

#define HIP_CHECK(expr) do { hipError_t _e = (expr); if (_e != hipSuccess) { \
    fprintf(stderr, "HIP error %s at %s:%d\n", hipGetErrorString(_e), __FILE__, __LINE__); \
    return; } } while (0)

typedef unsigned u32x4 __attribute__((ext_vector_type(4)));
typedef int v2i __attribute__((ext_vector_type(2)));
typedef int v4i __attribute__((ext_vector_type(4)));
typedef int v8i __attribute__((ext_vector_type(8)));
typedef float v4f __attribute__((ext_vector_type(4)));
#define LAS __attribute__((address_space(3)))
#define GAS __attribute__((address_space(1)))
#define CAS __attribute__((address_space(4)))
#define SB() __builtin_amdgcn_sched_barrier(0)

#define OUT_AUX 17
#define X_BITS "sc0 sc1"
#define WPF 1
#define SLEEP 1
#define RS0 10
#define DBG_ALLSTEPS 0
#define DBG_X 0
#define MAPR 8
#define COAL 1
#define XSTEP 65536
#define EXTRA 1
#define PFB 0
#define NOCONS 0
#define CDELAY 0
#define ZS0 1
#define XG 2

constexpr int NR = 8;
constexpr int MS = 512, KD = 4096, NC = 2048;
constexpr unsigned XB = MS * KD;
constexpr int NBLK = MS / 16;
constexpr int NPW = 16;
constexpr int GRID = 256;
constexpr int LW = 40960;
constexpr size_t OFF_CTR = 0;
constexpr size_t OFF_TMO = 1024 * 128;
constexpr size_t OFF_PCTR = OFF_TMO + 4096;
constexpr size_t OFF_FLAGS = OFF_PCTR + 4096;
constexpr unsigned PER = (NBLK + 1 - EXTRA) * NPW;
constexpr unsigned GATE = (NBLK - 2) * NPW;
constexpr unsigned NZW = 128;
constexpr size_t OFF_DBG = 256 << 10;
constexpr size_t OFF_STAGE = 1 << 20;
constexpr size_t WS_NEED = OFF_STAGE + 2 * (size_t)NR * XB;
constexpr unsigned long long DEADLINE = 40000000ull;

struct Params {
    char* dst[NR];
    const char* x; const char* w; const float* sx; const float* sw; float* out;
    unsigned rank; unsigned pad;
};

template <int N> __device__ __forceinline__ void waitv() { asm volatile("s_waitcnt vmcnt(%0)" :: "i"(N) : "memory"); }

__device__ __forceinline__ __amdgpu_buffer_rsrc_t mkrsrc(const void* p, unsigned bytes) {
    return __builtin_amdgcn_make_buffer_rsrc((void*)p, (short)0, (int)bytes, 0x00020000);
}

__device__ __forceinline__ unsigned ticket_issue(char* ctr) {
    unsigned t; unsigned long long sv;
    asm volatile("s_mov_b64 %1, exec\n\ts_mov_b64 exec, 1\n\tglobal_atomic_add %0, %2, %3, %4 sc0\n\ts_mov_b64 exec, %1"
                 : "=&v"(t), "=&s"(sv) : "v"(0), "v"(1), "s"(ctr) : "memory");
    return t;
}
__device__ __forceinline__ unsigned ticket_get(unsigned t) {
    asm volatile("" : "+v"(t));
    return __builtin_amdgcn_readfirstlane(t) + 1;
}

#define TS(slot) do { } while (0)

__global__ __launch_bounds__(256, 1) void ag_gemm_silu(Params p) {
    __shared__ __attribute__((aligned(16))) char lds[4 * LW];
    const int lane = threadIdx.x & 63;
    const int wid = __builtin_amdgcn_readfirstlane(threadIdx.x >> 6);
    const int bid = blockIdx.x;
    const int gwave = bid * 4 + wid;
    char* const me = p.dst[NR - 1];
    TS(0);
    constexpr int SL = 4 * XG;
    const int xw = (bid >> 3) - 4, gi = (bid & (XG - 1)) * 28 + xw;
    const int role = bid < 32 ? 0 : 1 + gi / SL, cs = ((bid & 7) / XG) * SL + gi % SL, pidx = bid;

    if (__builtin_expect(role == 0, 1)) {
        const int gw = pidx * 4 + wid;
        if (gw >= 7 * NPW) {
            const char* sb = p.x + (gw - 7 * NPW) * 4096;
            u32x4 t = {0u, 0u, 0u, 0u};
#pragma unroll
            for (int b = 1; b <= PFB; ++b) {
                sb += 65536;
#pragma unroll
                for (int j = 0; j < 4; ++j) asm volatile("global_load_dwordx4 %0, %1, %2 offset:%3" : "=v"(t) : "v"(lane * 16), "s"(sb), "i"(j * 1024) : "memory");
            }
            asm volatile("s_waitcnt vmcnt(0)" : "+v"(t) :: "memory");
            TS(48);
            return;
        }
        const int pi = gw % 7;
        const int vsrc = (lane & 15) * KD + (lane >> 4) * 16, vdst = lane * 16;
        u32x4 buf[4];
#define LOADB(i) asm volatile("global_load_dwordx4 %0, %1, %2 offset:%3" : "=v"(buf[i]) : "v"(vsrc), "s"(sb), "i"((i) * 64) : "memory")
#define STOREB(i) asm volatile("global_store_dwordx4 %0, %1, %2 offset:%3 sc0 sc1\n\ts_nop 1" :: "v"(vdst), "v"(buf[i]), "s"(db), "i"((i) * 1024) : "memory")
#define WAITB(n, i) asm volatile("s_waitcnt vmcnt(%2)" : "+v"(buf[i]), "+v"(tk) : "i"(n) : "memory")
#define DRAW(inc) asm volatile("s_mov_b64 exec, 1\n\tglobal_atomic_add_x2 %0, %1, %2, %3 sc0 sc1\n\ts_mov_b64 exec, -1" : "=&v"(tk) : "v"(0), "v"((unsigned long long)(inc)), "s"(ctr) : "memory")
#define FLAG() asm volatile("s_mov_b64 exec, 1\n\tglobal_store_dword %0, %1, %2 sc0 sc1\n\ts_mov_b64 exec, -1" :: "v"(0), "v"(call), "s"(fa) : "memory")
        const char* px = p.x; char* pw = p.dst[pi]; char* pm = me; unsigned rk = p.rank;
        asm volatile("" : "+s"(px), "+s"(pw), "+s"(pm), "+s"(rk));
        char* const ctr = pm + OFF_PCTR + pi * 128;
        unsigned u = gw / 7;
        u32x4 ext[EXTRA + 1][4];
        v2i tk;
        const char* sb = px + u * 256;
#pragma unroll
        for (int e = 0; e < EXTRA; ++e, sb += XSTEP) {
#pragma unroll
            for (int i = 0; i < 4; ++i) asm volatile("global_load_dwordx4 %0, %1, %2 offset:%3" : "=v"(ext[e][i]) : "v"(COAL ? (lane >> 2) * KD + (lane & 3) * 16 : vsrc), "s"(sb), "i"(i * 64) : "memory");
            if (e == 0) DRAW(1);
        }
        LOADB(0); LOADB(1); LOADB(2); LOADB(3);
        if (!EXTRA) DRAW(1);
#define FIXUP(e) do { asm volatile("" : "+v"(ext[e][0]), "+v"(ext[e][1]), "+v"(ext[e][2]), "+v"(ext[e][3])); \
        if (COAL) { _Pragma("unroll") for (int i = 0; i < 16; ++i) \
            ext[e][i >> 2][i & 3] = (unsigned)__builtin_amdgcn_ds_bpermute(((lane & 15) * 4 + (lane >> 4)) * 4, (int)ext[e][i >> 2][i & 3]); \
            asm volatile("" : "+v"(ext[e][0]), "+v"(ext[e][1]), "+v"(ext[e][2]), "+v"(ext[e][3])); } } while (0)
        if (EXTRA) { waitv<EXTRA * 4 + 1>(); FIXUP(0); }
        WAITB(EXTRA * 4, 0);
        const unsigned call = (unsigned)__builtin_amdgcn_readfirstlane(tk[0]) / PER + 1;
        const unsigned tbase = (call - 1) * PER - (EXTRA + 1) * NPW;
        char* const stage = pw + OFF_STAGE + ((size_t)(call & 1) * NR + rk) * XB;
        char* const flags = pw + OFF_FLAGS + rk * NPW * 128;
        char* fa = EXTRA ? flags + u * 128 : pm + OFF_TMO + 2048;
        int more = EXTRA - 1;
#pragma unroll
        for (int e = 0; e < EXTRA; ++e, u += NPW) {
            if (e) { waitv<EXTRA * 4>(); FIXUP(e); }
#pragma unroll
            for (int i = 0; i < 4; ++i)
                asm volatile("global_store_dwordx4 %0, %1, %2 offset:%3 sc0 sc1\n\ts_nop 1" :: "v"(vdst), "v"(ext[e][i]), "s"(stage + u * 4096), "i"(i * 1024) : "memory");
        }
        WAITB(EXTRA * 4, 0);
        TS(1);
        while (true) {
            WAITB(7, 0);
            const unsigned un = __builtin_amdgcn_readfirstlane(tk[0]) - tbase, ul = un < NBLK * NPW ? un : 0;
            if (__builtin_expect(!NOCONS && un >= GATE && (unsigned)__builtin_amdgcn_readfirstlane(tk[1]) < call * NZW, 0)) {
                const unsigned long long t0 = __builtin_amdgcn_s_memrealtime();
                do {
                    if (__builtin_amdgcn_s_memrealtime() - t0 > DEADLINE) {
                        __hip_atomic_store((unsigned*)(pm + OFF_TMO), call, __ATOMIC_RELAXED, __HIP_MEMORY_SCOPE_AGENT); return; }
                    __builtin_amdgcn_s_sleep(8);
                    DRAW(0); WAITB(0, 0);
                } while ((unsigned)__builtin_amdgcn_readfirstlane(tk[1]) < call * NZW);
            }
            DRAW(1);
            char* const db = stage + u * 4096;
            sb = px + (ul >> 4) * 65536 + (ul & 15) * 256;
            STOREB(0); LOADB(0);
            WAITB(8, 1); STOREB(1); LOADB(1);
            WAITB(8, 2); STOREB(2); LOADB(2);
            WAITB(7, 3);
            FLAG();
            if (EXTRA > 1) for (; more > 0; --more) { fa += 8; FLAG(); }
            STOREB(3); LOADB(3);
            fa = flags + (u & 15) * 128 + ((u >> 4) & 15) * 8 + (u >> 8) * 4;
            if (DBG_ALLSTEPS) TS(3 + (u >> 4));
            u = un;
            if (u >= NBLK * NPW) break;
        }
        WAITB(0, 0);
        FLAG();
        asm volatile("" :: "v"(buf[1]), "v"(buf[2]), "v"(buf[3]));
        TS(2);
        return;
    }

    if (NOCONS) { if (p.sx[0] != 123.f) return; }
    if (CDELAY) { const unsigned long long t0 = __builtin_amdgcn_s_memrealtime(); while (__builtin_amdgcn_s_memrealtime() - t0 < CDELAY) __builtin_amdgcn_s_sleep(2); }
    const unsigned srcr = (p.rank + role) % NR;
    const int g = lane >> 4, li = lane & 15, sub = cs & 3;
    char* const myl = lds + wid * LW;
    const unsigned lbase = (unsigned)(size_t)(LAS char*)myl;
    const unsigned tk = ticket_issue(me + OFF_CTR + (size_t)gwave * 128);
    const __amdgpu_buffer_rsrc_t rs_w = mkrsrc(p.w, (unsigned)(KD * NC));
    constexpr int NPF = (SL * 2048 + 112 * 64 - 1) / (112 * 64);
    unsigned pf[NPF];
    if (WPF) {
#pragma unroll
        for (int k = 0; k < NPF; ++k) {
            const int l = ((k * 28 + xw) * 4 + wid) * 64 + lane;
            asm volatile("buffer_load_dword %0, %1, %2, 0 offen" : "=v"(pf[k]) : "v"((l / (SL / 2)) * NC + (cs / SL) * SL * 64 + (l % (SL / 2)) * 128), "s"(rs_w) : "memory");
        }
    }
    u32x4 xb[2][16];
    v8i wf[8][4];
    const bool dummy = role >= 5;
#define OWN_SO(j) (((j) == 0 ? (dummy ? NBLK - 1 : 27 + role) : 7 * (((j) - 1 + sub) & 3) + role - 1) * 65536)
    int vrow[16];
#pragma unroll
    for (int r = 0; r < 16; ++r) vrow[r] = r * KD + wid * 1024 + ((lane ^ r) * 16);
    __amdgpu_buffer_rsrc_t rs_own = mkrsrc(p.x, XB);
    int so_d;
#define DMA(bf, r) asm volatile("s_add_u32 m0, %0, %1\n\ts_nop 0\n\tbuffer_load_dwordx4 %2, %3, %4 offen lds" \
        :: "s"(lbase), "i"(8192 + (bf) * 16384 + (r) * 1024), "v"(vrow[r]), "s"(rs_own), "s"(so_d) : "memory", "scc")
    int xa[4];
#pragma unroll
    for (int j = 0; j < 4; ++j) xa[j] = li * 1024 + (((4 * j + g) ^ li) * 16);
#define XRD(bf, q_) xb[bf][q_] = *(LAS u32x4*)((LAS char*)myl + 8192 + (bf) * 16384 + ((q_) >> 2) * 256 + xa[(q_) & 3])

    {
        const int kr = lane >> 2;
        const int wvA = (wid * 1024 + kr) * NC + cs * 64 + (((lane & 3) ^ ((kr >> 2) & 1)) * 16);
        const int wvB = wvA ^ 32;
#define WLOAD(r) do { _Pragma("unroll") for (int j = 0; j < 16; ++j) \
            asm volatile("s_add_u32 m0, %0, %1\n\ts_nop 0\n\tbuffer_load_dwordx4 %2, %3, %4 offen lds" \
                :: "s"(lbase), "i"(8192 + ((r) & 1) * 16384 + j * 1024), "v"((j & 1) ? wvB : wvA), "s"(rs_w), "s"(((r) * 256 + j * 16) * NC) : "memory", "scc"); } while (0)
        const int q = li >> 1, h = li & 1;
        const int f0 = ((q >> 2) & 1) | ((g & 1) << 1);
        const int rbase = (16 * g + q) * 64 + 8 * h;
#define WREAD(r) do { _Pragma("unroll") for (int ks = 0; ks < 2; ++ks) _Pragma("unroll") for (int t = 0; t < 4; ++t) { \
            v2i e[4]; \
            _Pragma("unroll") for (int u = 0; u < 4; ++u) \
                e[u] = __builtin_amdgcn_ds_read_tr8_b64_v2i32((LAS v2i*)(myl + 8192 + ((r) & 1) * 16384 + ks * 8192 + (u >> 1) * 4096 + (u & 1) * 512 + rbase + ((t ^ f0) * 16))); \
            wf[(r) * 2 + ks][t] = (v8i){e[0][0], e[0][1], e[1][0], e[1][1], e[2][0], e[2][1], e[3][0], e[3][1]}; } \
            asm volatile("s_waitcnt lgkmcnt(0)" ::: "memory"); } while (0)
        WLOAD(0); WLOAD(1);
        waitv<16>();
        if (WPF) {
#pragma unroll
            for (int k = 0; k < NPF; ++k) asm volatile("" :: "v"(pf[k]));
        }
        WREAD(0); WLOAD(2);
        waitv<16>(); WREAD(1); WLOAD(3);
        waitv<16>(); WREAD(2);
        so_d = OWN_SO(0);
#pragma unroll
        for (int r = 0; r < 16; ++r) DMA(0, r);
        waitv<16>(); WREAD(3);
        so_d = OWN_SO(1);
#pragma unroll
        for (int r = 0; r < 16; ++r) DMA(1, r);
        waitv<16>();
#pragma unroll
        for (int q2 = 0; q2 < 16; ++q2) XRD(0, q2);
    }
    TS(1);
    const float scale = *(const CAS float*)p.sx * *(const CAS float*)p.sw;
    const unsigned call = ticket_get(tk);
    const char* const stg = me + OFF_STAGE + ((size_t)(call & 1) * NR + srcr) * XB;
    const char* const zslot = me + OFF_STAGE + ((size_t)(~call & 1) * NR + srcr) * XB;
    const int vo_z = (cs * 4 + wid) * 16384 + lane * 16;
    u32x4 zero = {0u, 0u, 0u, 0u};
    asm volatile("" : "+v"(zero));
#define ZERO(n) do { int n_ = (n) - ZS0; asm volatile("" : "+s"(n_)); const __amdgpu_buffer_rsrc_t rz = mkrsrc(zslot, (unsigned)n_ < 16u ? XB : 0); \
        asm volatile("buffer_store_dwordx4 %0, %1, %2, %3 offen sc0 sc1" :: "v"(zero), "v"(vo_z), "s"(rz), "s"(n_ * 1024) : "memory"); } while (0)
#define ZDONE() asm volatile("s_mov_b64 exec, 1\n\tglobal_atomic_add_x2 %0, %1, %2 sc1\n\ts_mov_b64 exec, -1\n\ts_nop 1" :: "v"(0), "v"(1ull << 32), "s"(me + OFF_PCTR + (role - 1) * 128) : "memory")
    const __amdgpu_buffer_rsrc_t rs_out = mkrsrc(p.out, (unsigned)(NR * MS * NC * 4));
    const int vo_x = lane * 16 + wid * 16384;
    unsigned* const tmo = (unsigned*)(me + OFF_TMO);
    const char* const pollb = me + OFF_FLAGS + (srcr * NPW + wid * 4) * 128;
    const int pollv = lane * 8;
    v2i pv = {0, 0};
    v4f cin = {0.f, 0.f, 0.f, 0.f};
    const int wr_off = g * 256 + ((li ^ g) * 16);
    const int mrow = 4 * wid + g;
    const int rd_off = (li >> 2) * 1024 + (li & 3) * 256 + ((mrow ^ li) * 16);
    const int vo_out = mrow * (NC * 4) + cs * 256 + li * 16;
    int vo_st = (int)0x80000000u, vo_nx = dummy ? (int)0x80000000u : vo_out;
    v4f accA[4], accB[4], part[4], res = {0.f, 0.f, 0.f, 0.f}, ex;
#pragma unroll
    for (int t = 0; t < 4; ++t) accB[t] = (v4f){0.f, 0.f, 0.f, 0.f};

    const int oo_own = (int)p.rank * (int)XB * 2, oo_rem = (int)srcr * (int)XB * 2;
    int oo_p = 0, oo_c = oo_own + 2 * OWN_SO(0), oo_n = oo_own + 2 * OWN_SO(1);
    __amdgpu_buffer_rsrc_t rs_;
    int so_q; unsigned sp_c = 0, sp_1 = 0, sp_2 = 0;
    unsigned known = 0;
#define DECODE(pv_) do { const unsigned long long m0 = __builtin_amdgcn_ballot_w64((unsigned)pv_[0] >= call), m1 = __builtin_amdgcn_ballot_w64((unsigned)pv_[1] >= call); \
        unsigned a0 = (unsigned)m0 & (unsigned)(m0 >> 32), a1 = (unsigned)m1 & (unsigned)(m1 >> 32); a0 &= a0 >> 16; a1 &= a1 << 16; \
        known |= (a0 & 0xffffu) | (a1 & 0xffff0000u); asm volatile("" : "+s"(known)); } while (0)
#define POLL(pv_) asm volatile("global_load_dwordx2 %0, %1, %2 sc0 sc1" : "+v"(pv_) : "v"(pollv), "s"(pollb) : "memory")
#define DBGW0() do { } while (0)
#define DBGW() do { } while (0)
#define DBGR() do { } while (0)
#define LD(bf, q_) do { asm volatile("buffer_load_dwordx4 %0, %1, %2, %3 offen offset:%4 " X_BITS : "=v"(xb[bf][q_]) : "v"(vo_x), "s"(rs_), "s"(so_q), "i"(((q_) & 3) * 1024) : "memory"); \
        if (((q_) & 3) == 3 && (q_) < 15) { asm volatile("s_add_u32 %0, %0, 0x1000" : "+s"(so_q) :: "scc"); } } while (0)
#define PLAN(n) do { int n_ = (n); asm volatile("" : "+s"(n_)); \
        sp_2 = n_ < NBLK ? (1u << n_) & ~known : 0u; so_q = n_ * 65536; \
        rs_ = mkrsrc(stg, n_ < NBLK ? XB : 0); asm volatile("" : "+s"(sp_2), "+s"(so_q)); } while (0)
    unsigned zm0, zm1;
#define ZMIN(bf, q) do { asm volatile("" : "+v"(xb[bf][q]), "+v"(xb[bf][(q) + 1])); \
        zm0 = min(min(xb[bf][q][0], xb[bf][q][1]), (q) == 0 ? ~0u : zm0); zm1 = min(min(xb[bf][q][2], xb[bf][q][3]), (q) == 0 ? ~0u : zm1); \
        zm0 = min(min(xb[bf][(q) + 1][0], xb[bf][(q) + 1][1]), zm0); zm1 = min(min(xb[bf][(q) + 1][2], xb[bf][(q) + 1][3]), zm1); \
        asm volatile("" : "+v"(zm0), "+v"(zm1)); } while (0)
#define ZTEST(sp) do { if (__builtin_amdgcn_ballot_w64(min(zm0, zm1) == 0) == 0) sp = 0; asm volatile("" : "+s"(sp)); } while (0)
#define RELOAD(bf, c) do { so_q = (c) * 65536; _Pragma("unroll") for (int q = 0; q < 16; ++q) LD(bf, q); } while (0)
#define RETRY(bf, c) do { \
        unsigned long long t0 = 0; \
        DBGW0(); TS(43); DBGR(); \
        DECODE(pv); \
        rs_ = mkrsrc(stg, XB); \
        do { \
            unsigned tm; \
            sp_c &= ~known; \
            RELOAD(bf, c); \
            POLL(pv); \
            asm volatile("global_load_dword %1, %2, %3 sc1\n\ts_waitcnt vmcnt(0)" : "+v"(pv), "=&v"(tm) : "v"(0), "s"(tmo) : "memory"); \
            DECODE(pv); \
            _Pragma("unroll") for (int q = 0; q < 16; q += 2) ZMIN(bf, q); \
            ZTEST(sp_c); \
            const unsigned long long t1 = __builtin_amdgcn_s_memrealtime(); if (t0 == 0) t0 = t1; \
            if (sp_c && ((unsigned)__builtin_amdgcn_readfirstlane(tm) == call || t1 - t0 > DEADLINE)) { \
                __hip_atomic_store(tmo, call, __ATOMIC_RELAXED, __HIP_MEMORY_SCOPE_AGENT); \
                known = ~0u; sp_c = 0; cin = (v4f){__builtin_nanf(""), __builtin_nanf(""), __builtin_nanf(""), __builtin_nanf("")}; \
                asm volatile("s_nop 3" : "+v"(cin)); } \
            if (SLEEP) __builtin_amdgcn_s_sleep(SLEEP); \
        } while (sp_c); \
        if (sp_1) { sp_1 &= ~known; RELOAD((bf) ^ 1, (c) + 1); POLL(pv); POLL(pv); } \
        TS(44); DBGW(); } while (0)
#define RED_WRITE(acc, bf, t) *(LAS v4f*)((LAS char*)myl + (bf) * 4096 + (t) * 1024 + (wr_off ^ ((t) * 64))) = acc[t]
#define RED_BAR() asm volatile("s_waitcnt lgkmcnt(0)\n\ts_barrier" ::: "memory")
#define RED_READ(bf, w) part[w] = *(LAS v4f*)((LAS char*)lds + (w) * LW + (bf) * 4096 + rd_off)
#define ADD4(a, b) do { _Pragma("unroll") for (int r_ = 0; r_ < 4; ++r_) { a[r_] += b[r_]; asm volatile("" : "+v"(a[r_])); } } while (0)
#define EPI_A(r) do { res[r] = part[0][r] * scale; \
        ex[r] = __builtin_amdgcn_exp2f(__builtin_amdgcn_fmed3f(res[r], -60.f, 60.f) * -1.44269504088896f); asm volatile("" : "+v"(res[r]), "+v"(ex[r])); } while (0)
#define EPI_B(r) do { res[r] *= __builtin_amdgcn_rcpf(1.f + ex[r]); asm volatile("" : "+v"(res[r])); } while (0)
#define STORE_RES(vo, so) __builtin_amdgcn_raw_buffer_store_b128(__builtin_bit_cast(u32x4, res), rs_out, vo, so, OUT_AUX)
#define MF(acc, t, ks) do { if (ks == 0) asm volatile("v_mfma_f32_16x16x128_f8f6f4 %0, %1, %2, %3 cbsz:1" : "=&v"(acc[t]) : "a"(wf[ks][t]), "v"(xv), "v"(cin)); \
        else asm volatile("v_mfma_f32_16x16x128_f8f6f4 %0, %1, %2, %0 cbsz:1" : "+v"(acc[t]) : "a"(wf[ks][t]), "v"(xv)); } while (0)
#define FILL(s, bf, accp, vo) do { \
        if (!(DBG_X & 1)) { \
        if (s == 1 || s == 3 || s == 5 || s == 7) RED_WRITE(accp, (bf) ^ 1, s >> 1); \
        if (s == RS0) RED_BAR(); \
        if (s == RS0 || s == RS0 + 1) { RED_READ((bf) ^ 1, 2 * (s - RS0)); RED_READ((bf) ^ 1, 2 * (s - RS0) + 1); } \
        if (s == RS0 + 4) ADD4(part[0], part[1]); \
        if (s == RS0 + 5) ADD4(part[2], part[3]); \
        if (s == RS0 + 6) ADD4(part[0], part[2]); } \
        if (!(DBG_X & 2)) { \
        if (s >= RS0 + 7 && s < RS0 + 11) EPI_A(s - RS0 - 7); \
        if (s >= RS0 + 11 && s < RS0 + 15) EPI_B(s - RS0 - 11); } \
        if (s == RS0 + 16 && !(DBG_X & 4)) STORE_RES(vo, oo_p); \
        } while (0)
#define XV(bf, ks) const u32x4 lo_ = xb[bf][2 * ks], hi_ = xb[bf][2 * ks + 1]; \
        const v8i xv = {(int)lo_[0], (int)lo_[1], (int)lo_[2], (int)lo_[3], (int)hi_[0], (int)hi_[1], (int)hi_[2], (int)hi_[3]}
#define OSTEP(j, bf, acc, accp) do { \
        so_d = OWN_SO((j) + 2); if ((bf) == 1) rs_own = mkrsrc(p.x, (j) < 3 ? XB : 0); \
        asm volatile("s_waitcnt lgkmcnt(0)" ::: "memory"); SB(); \
        _Pragma("unroll") for (int ks = 0; ks < 8; ++ks) { XV(bf, ks); \
            _Pragma("unroll") for (int t = 0; t < 4; ++t) { const int sl = 4 * ks + t; \
                MF(acc, t, ks); SB(); \
                if (sl == 14) waitv<15>(); \
                if (sl < 16) DMA(bf, sl); \
                if (sl >= 14 && sl < 30) XRD((bf) ^ 1, sl - 14); \
                FILL(sl, bf, accp, vo_st); \
                if (sl == 30) { vo_st = vo_nx; vo_nx = vo_out; asm volatile("" : "+v"(vo_st), "+v"(vo_nx)); } \
                if (sl == 1 && (bf) == 1) PLAN(0); \
                if (t == 3 && (bf) == 1 && (j) == 3) { LD(1, 2 * ks); LD(1, 2 * ks + 1); } \
                SB(); } } \
        oo_p = oo_c; oo_c = oo_n; oo_n = oo_own + 2 * so_d; asm volatile("" : "+s"(oo_p), "+s"(oo_c), "+s"(oo_n)); \
        if (DBG_ALLSTEPS) TS(2 + (j)); \
        } while (0)
#define GSTEP(n, bf, acc, accp) do { \
        sp_c = sp_1; sp_1 = sp_2; \
        if (__builtin_expect(sp_c != 0, 0)) RETRY(bf, (n) - 2); \
        SB(); \
        _Pragma("unroll") for (int ks = 0; ks < 8; ++ks) { \
              \
            asm volatile("s_waitcnt vmcnt(18)" : "+v"(xb[bf][2 * ks]), "+v"(xb[bf][2 * ks + 1]) :: "memory"); \
            XV(bf, ks); \
            _Pragma("unroll") for (int t = 0; t < 4; ++t) { const int sl = 4 * ks + t; \
                MF(acc, t, ks); SB(); \
                FILL(sl, bf, accp, vo_out); \
                if (sl == 0) PLAN(n); \
                if (sl == 2 && (bf) == 1 && (n) == ((ZS0 + 18) & ~1)) ZDONE(); \
                if (sl == 24) ZERO(n); \
                if (sl == 4 || sl == 8 || sl == 12) ZMIN((bf) ^ 1, sl / 2 - 2); \
                if (sl == 25) ZMIN((bf) ^ 1, 6); \
                if (sl >= 27 && sl < 30) ZMIN((bf) ^ 1, 2 * sl - 46); \
                if (sl == 30) { asm volatile("s_waitcnt vmcnt(16)" : "+v"(pv) :: "memory"); ZMIN((bf) ^ 1, 14); } \
                if (sl == 31) ZTEST(sp_1); \
                if (sl == 8) { oo_n = (n) - 1; asm volatile("" : "+s"(oo_n)); oo_n = oo_rem + 131072 * oo_n; asm volatile("" : "+s"(oo_n)); } \
                if (t == 3) { LD(bf, 2 * ks); LD(bf, 2 * ks + 1); } \
                SB(); } } \
        oo_p = oo_c; oo_c = oo_n; asm volatile("" : "+s"(oo_p), "+s"(oo_c)); \
        if (DBG_ALLSTEPS) TS(5 + (n)); else if ((n) == 16) TS(21); \
        } while (0)

    POLL(pv);
    for (int j = 0; j < 4; j += 2) { OSTEP(j, 0, accA, accB); OSTEP(j + 1, 1, accB, accA); }
    if (!DBG_ALLSTEPS) TS(2);
    for (int n = 0; n <= NBLK; n += 2) { if (n > 0) GSTEP(n, 1, accB, accA); GSTEP(n + 1, 0, accA, accB); }
    if (!DBG_ALLSTEPS) TS(39);
    asm volatile("s_nop 15" : "+v"(accA[0]), "+v"(accA[1]), "+v"(accA[2]), "+v"(accA[3]));
    RED_WRITE(accA, 0, 0); RED_WRITE(accA, 0, 1); RED_WRITE(accA, 0, 2); RED_WRITE(accA, 0, 3);
    RED_BAR(); RED_READ(0, 0); RED_READ(0, 1); RED_READ(0, 2); RED_READ(0, 3);
    ADD4(part[0], part[1]); ADD4(part[2], part[3]); ADD4(part[0], part[2]);
    EPI_A(0); EPI_A(1); EPI_A(2); EPI_A(3); EPI_B(0); EPI_B(1); EPI_B(2); EPI_B(3);
    STORE_RES(vo_out, oo_p);
    SB();
    asm volatile("s_waitcnt vmcnt(0)" : "+v"(pv) :: "memory");
#pragma unroll
    for (int q = 0; q < 16; ++q) asm volatile("" :: "v"(xb[0][q]), "v"(xb[1][q]));
}

extern "C" void kernel_launch(int rank, int nranks, ncclComm_t comm, hipStream_t stream,
                              void* const* d_in, const int* in_sizes, int n_in,
                              void* d_out, int out_size, const int* in_dtypes, int out_dtype,
                              void* const* d_out_all, void* const* d_ws_all, size_t ws_bytes) {
    (void)comm; (void)in_dtypes; (void)out_dtype; (void)d_out_all;
    if (nranks != NR || rank < 0 || rank >= NR || n_in < 4 || ws_bytes < WS_NEED || in_sizes[0] != (int)XB || in_sizes[1] != KD * NC
        || out_size != NR * MS * NC) { fprintf(stderr, "unexpected problem shape\n"); return; }
    Params p;
    for (int i = 0; i < NR; ++i) p.dst[i] = (char*)d_ws_all[(rank + 1 + i) % NR];
    p.x = (const char*)d_in[0]; p.w = (const char*)d_in[1]; p.sx = (const float*)d_in[2]; p.sw = (const float*)d_in[3];
    p.out = (float*)d_out; p.rank = (unsigned)rank; p.pad = 0;
    ag_gemm_silu<<<GRID, 256, 0, stream>>>(p);
    HIP_CHECK(hipGetLastError());
}
```

```cpp
#include <hip/hip_runtime.h>
#include <rccl/rccl.h>
#include <cstdio>
#include <cstdint>

#define HIP_CHECK(expr) do { hipError_t _e = (expr); if (_e != hipSuccess) { \
    fprintf(stderr, "HIP error %s at %s:%d\n", hipGetErrorString(_e), __FILE__, __LINE__); \
    return; } } while (0)

typedef unsigned u32x4 __attribute__((ext_vector_type(4)));
typedef int v2i __attribute__((ext_vector_type(2)));
typedef int v4i __attribute__((ext_vector_type(4)));
typedef int v8i __attribute__((ext_vector_type(8)));
typedef float v4f __attribute__((ext_vector_type(4)));
typedef float v2f __attribute__((ext_vector_type(2)));
#define LAS __attribute__((address_space(3)))
#define GAS __attribute__((address_space(1)))
#define CAS __attribute__((address_space(4)))
#define SB() __builtin_amdgcn_sched_barrier(0)

#define OUT_AUX 17
#define X_BITS "sc0 sc1"
#define WPF 1
#define SLEEP 1
#define RS0 10
#define DBG_ALLSTEPS 0
#define DBG_X 0
#define MAPR 8
#define COAL 1
#define XSTEP 65536
#define EXTRA 1
#define PFB 0
#define NOCONS 0
#define CDELAY 0
#define ZS0 1
#define XG 2

constexpr int NR = 8;
constexpr int MS = 512, KD = 4096, NC = 2048;
constexpr unsigned XB = MS * KD;
constexpr int NBLK = MS / 16;
constexpr int NPW = 16;
constexpr int GRID = 256;
constexpr int LW = 40960;
constexpr size_t OFF_CTR = 0;
constexpr size_t OFF_TMO = 1024 * 128;
constexpr size_t OFF_PCTR = OFF_TMO + 4096;
constexpr size_t OFF_FLAGS = OFF_PCTR + 4096;
constexpr unsigned PER = (NBLK + 1 - EXTRA) * NPW;
constexpr unsigned GATE = (NBLK - 2) * NPW;
constexpr unsigned NZW = 32;
constexpr size_t OFF_DBG = 256 << 10;
constexpr size_t OFF_STAGE = 1 << 20;
constexpr size_t WS_NEED = OFF_STAGE + 2 * (size_t)NR * XB;
constexpr unsigned long long DEADLINE = 40000000ull;

struct Params {
    char* dst[NR];
    const char* x; const char* w; const float* sx; const float* sw; float* out;
    unsigned rank; unsigned pad;
};

template <int N> __device__ __forceinline__ void waitv() { asm volatile("s_waitcnt vmcnt(%0)" :: "i"(N) : "memory"); }

__device__ __forceinline__ __amdgpu_buffer_rsrc_t mkrsrc(const void* p, unsigned bytes) {
    return __builtin_amdgcn_make_buffer_rsrc((void*)p, (short)0, (int)bytes, 0x00020000);
}

__device__ __forceinline__ unsigned ticket_issue(char* ctr) {
    unsigned t; unsigned long long sv;
    asm volatile("s_mov_b64 %1, exec\n\ts_mov_b64 exec, 1\n\tglobal_atomic_add %0, %2, %3, %4 sc0\n\ts_mov_b64 exec, %1"
                 : "=&v"(t), "=&s"(sv) : "v"(0), "v"(1), "s"(ctr) : "memory");
    return t;
}
__device__ __forceinline__ unsigned ticket_get(unsigned t) {
    asm volatile("" : "+v"(t));
    return __builtin_amdgcn_readfirstlane(t) + 1;
}

#define TS(slot) do { } while (0)

__global__ __launch_bounds__(256, 1) void ag_gemm_silu(Params p) {
    __shared__ __attribute__((aligned(16))) char lds[4 * LW];
    const int lane = threadIdx.x & 63;
    const int wid = __builtin_amdgcn_readfirstlane(threadIdx.x >> 6);
    const int bid = blockIdx.x;
    const int gwave = bid * 4 + wid;
    char* const me = p.dst[NR - 1];
    TS(0);
    constexpr int SL = 4 * XG;
    const int xw = (bid >> 3) - 4, gi = (bid & (XG - 1)) * 28 + xw;
    const int role = bid < 32 ? 0 : 1 + gi / SL, cs = ((bid & 7) / XG) * SL + gi % SL, pidx = bid;

    if (__builtin_expect(role == 0, 1)) {
        const int gw = pidx * 4 + wid;
        if (gw >= 7 * NPW) {
            const char* sb = p.x + (gw - 7 * NPW) * 4096;
            u32x4 t = {0u, 0u, 0u, 0u};
#pragma unroll
            for (int b = 1; b <= PFB; ++b) {
                sb += 65536;
#pragma unroll
                for (int j = 0; j < 4; ++j) asm volatile("global_load_dwordx4 %0, %1, %2 offset:%3" : "=v"(t) : "v"(lane * 16), "s"(sb), "i"(j * 1024) : "memory");
            }
            asm volatile("s_waitcnt vmcnt(0)" : "+v"(t) :: "memory");
            TS(48);
            return;
        }
        const int pi = gw % 7;
        const int vsrc = (lane & 15) * KD + (lane >> 4) * 16, vdst = lane * 16;
        u32x4 buf[4];
#define LOADB(i) asm volatile("global_load_dwordx4 %0, %1, %2 offset:%3" : "=v"(buf[i]) : "v"(vsrc), "s"(sb), "i"((i) * 64) : "memory")
#define STOREB(i) asm volatile("global_store_dwordx4 %0, %1, %2 offset:%3 sc0 sc1\n\ts_nop 1" :: "v"(vdst), "v"(buf[i]), "s"(db), "i"((i) * 1024) : "memory")
#define WAITB(n, i) asm volatile("s_waitcnt vmcnt(%2)" : "+v"(buf[i]), "+v"(tk) : "i"(n) : "memory")
#define DRAW(inc) asm volatile("s_mov_b64 exec, 1\n\tglobal_atomic_add_x2 %0, %1, %2, %3 sc0 sc1\n\ts_mov_b64 exec, -1" : "=&v"(tk) : "v"(0), "v"((unsigned long long)(inc)), "s"(ctr) : "memory")
#define FLAG() asm volatile("s_mov_b64 exec, 1\n\tglobal_store_dword %0, %1, %2 sc0 sc1\n\ts_mov_b64 exec, -1" :: "v"(0), "v"(call), "s"(fa) : "memory")
        const char* px = p.x; char* pw = p.dst[pi]; char* pm = me; unsigned rk = p.rank;
        asm volatile("" : "+s"(px), "+s"(pw), "+s"(pm), "+s"(rk));
        char* const ctr = pm + OFF_PCTR + pi * 128;
        unsigned u = gw / 7;
        u32x4 ext[EXTRA + 1][4];
        v2i tk;
        const char* sb = px + u * 256;
#pragma unroll
        for (int e = 0; e < EXTRA; ++e, sb += XSTEP) {
#pragma unroll
            for (int i = 0; i < 4; ++i) asm volatile("global_load_dwordx4 %0, %1, %2 offset:%3" : "=v"(ext[e][i]) : "v"(COAL ? (lane >> 2) * KD + (lane & 3) * 16 : vsrc), "s"(sb), "i"(i * 64) : "memory");
            if (e == 0) DRAW(1);
        }
        LOADB(0); LOADB(1); LOADB(2); LOADB(3);
        if (!EXTRA) DRAW(1);
#define FIXUP(e) do { asm volatile("" : "+v"(ext[e][0]), "+v"(ext[e][1]), "+v"(ext[e][2]), "+v"(ext[e][3])); \
        if (COAL) { _Pragma("unroll") for (int i = 0; i < 16; ++i) \
            ext[e][i >> 2][i & 3] = (unsigned)__builtin_amdgcn_ds_bpermute(((lane & 15) * 4 + (lane >> 4)) * 4, (int)ext[e][i >> 2][i & 3]); \
            asm volatile("" : "+v"(ext[e][0]), "+v"(ext[e][1]), "+v"(ext[e][2]), "+v"(ext[e][3])); } } while (0)
        if (EXTRA) { waitv<EXTRA * 4 + 1>(); FIXUP(0); }
        WAITB(EXTRA * 4, 0);
        const unsigned call = (unsigned)__builtin_amdgcn_readfirstlane(tk[0]) / PER + 1;
        const unsigned tbase = (call - 1) * PER - (EXTRA + 1) * NPW;
        char* const stage = pw + OFF_STAGE + ((size_t)(call & 1) * NR + rk) * XB;
        char* const flags = pw + OFF_FLAGS + rk * NPW * 128;
        char* fa = EXTRA ? flags + u * 128 : pm + OFF_TMO + 2048;
        int more = EXTRA - 1;
#pragma unroll
        for (int e = 0; e < EXTRA; ++e, u += NPW) {
            if (e) { waitv<EXTRA * 4>(); FIXUP(e); }
#pragma unroll
            for (int i = 0; i < 4; ++i)
                asm volatile("global_store_dwordx4 %0, %1, %2 offset:%3 sc0 sc1\n\ts_nop 1" :: "v"(vdst), "v"(ext[e][i]), "s"(stage + u * 4096), "i"(i * 1024) : "memory");
        }
        WAITB(EXTRA * 4, 0);
        TS(1);
        while (true) {
            WAITB(7, 0);
            const unsigned un = __builtin_amdgcn_readfirstlane(tk[0]) - tbase, ul = un < NBLK * NPW ? un : 0;
            if (__builtin_expect(!NOCONS && un >= GATE && (unsigned)__builtin_amdgcn_readfirstlane(tk[1]) < call * NZW, 0)) {
                const unsigned long long t0 = __builtin_amdgcn_s_memrealtime();
                do {
                    if (__builtin_amdgcn_s_memrealtime() - t0 > DEADLINE) {
                        __hip_atomic_store((unsigned*)(pm + OFF_TMO), call, __ATOMIC_RELAXED, __HIP_MEMORY_SCOPE_AGENT); return; }
                    __builtin_amdgcn_s_sleep(8);
                    DRAW(0); WAITB(0, 0);
                } while ((unsigned)__builtin_amdgcn_readfirstlane(tk[1]) < call * NZW);
            }
            DRAW(1);
            char* const db = stage + u * 4096;
            sb = px + (ul >> 4) * 65536 + (ul & 15) * 256;
            STOREB(0); LOADB(0);
            WAITB(8, 1); STOREB(1); LOADB(1);
            WAITB(8, 2); STOREB(2); LOADB(2);
            WAITB(7, 3);
            FLAG();
            if (EXTRA > 1) for (; more > 0; --more) { fa += 8; FLAG(); }
            STOREB(3); LOADB(3);
            fa = flags + (u & 15) * 128 + ((u >> 4) & 15) * 8 + (u >> 8) * 4;
            if (DBG_ALLSTEPS) TS(3 + (u >> 4));
            u = un;
            if (u >= NBLK * NPW) break;
        }
        WAITB(0, 0);
        FLAG();
        asm volatile("" :: "v"(buf[1]), "v"(buf[2]), "v"(buf[3]));
        TS(2);
        return;
    }

    if (NOCONS) { if (p.sx[0] != 123.f) return; }
    if (CDELAY) { const unsigned long long t0 = __builtin_amdgcn_s_memrealtime(); while (__builtin_amdgcn_s_memrealtime() - t0 < CDELAY) __builtin_amdgcn_s_sleep(2); }
    const char* a_x = p.x; const char* a_w = p.w; const float* a_sx = p.sx; const float* a_sw = p.sw; float* a_out = p.out; unsigned a_rank = p.rank;
    asm volatile("" : "+s"(a_x), "+s"(a_w), "+s"(a_sx), "+s"(a_sw), "+s"(a_out), "+s"(a_rank));
    const unsigned srcr = (a_rank + role) % NR;
    const int g = lane >> 4, li = lane & 15, sub = cs & 3;
    char* const myl = lds + wid * LW;
    const unsigned lbase = (unsigned)(size_t)(LAS char*)myl;
    const unsigned tk = ticket_issue(me + OFF_CTR + (size_t)gwave * 128);
    const __amdgpu_buffer_rsrc_t rs_w = mkrsrc(a_w, (unsigned)(KD * NC));
    constexpr int NPF = (SL * 2048 + 112 * 64 - 1) / (112 * 64);
    unsigned pf[NPF];
    if (WPF) {
#pragma unroll
        for (int k = 0; k < NPF; ++k) {
            const int l = ((k * 28 + xw) * 4 + wid) * 64 + lane;
            asm volatile("buffer_load_dword %0, %1, %2, 0 offen" : "=v"(pf[k]) : "v"((l / (SL / 2)) * NC + (cs / SL) * SL * 64 + (l % (SL / 2)) * 128), "s"(rs_w) : "memory");
        }
    }
    float sxv, swv;
    u32x4 xb[2][16];
    v8i wf[8][4];
    const bool dummy = role >= 5;
#define OWN_SO(j) (((j) == 0 ? (dummy ? NBLK - 1 : 27 + role) : 7 * (((j) - 1 + sub) & 3) + role - 1) * 65536)
    int vrow[16];
#pragma unroll
    for (int r = 0; r < 16; ++r) vrow[r] = r * KD + wid * 1024 + ((lane ^ r) * 16);
    __amdgpu_buffer_rsrc_t rs_own = mkrsrc(a_x, XB);
    int so_d;
#define DMA(bf, r) asm volatile("s_add_u32 m0, %0, %1\n\ts_nop 0\n\tbuffer_load_dwordx4 %2, %3, %4 offen lds" \
        :: "s"(lbase), "i"(8192 + (bf) * 16384 + (r) * 1024), "v"(vrow[r]), "s"(rs_own), "s"(so_d) : "memory", "scc")
    int xa[4];
#pragma unroll
    for (int j = 0; j < 4; ++j) xa[j] = li * 1024 + (((4 * j + g) ^ li) * 16);
#define XRD(bf, q_) xb[bf][q_] = *(LAS u32x4*)((LAS char*)myl + 8192 + (bf) * 16384 + ((q_) >> 2) * 256 + xa[(q_) & 3])

    {
        const int kr = lane >> 2;
        const int wvA = (wid * 1024 + kr) * NC + cs * 64 + (((lane & 3) ^ ((kr >> 2) & 1)) * 16);
        const int wvB = wvA ^ 32;
#define WLOAD(r) do { _Pragma("unroll") for (int j = 0; j < 16; ++j) \
            asm volatile("s_add_u32 m0, %0, %1\n\ts_nop 0\n\tbuffer_load_dwordx4 %2, %3, %4 offen lds" \
                :: "s"(lbase), "i"(8192 + ((r) & 1) * 16384 + j * 1024), "v"((j & 1) ? wvB : wvA), "s"(rs_w), "s"(((r) * 256 + j * 16) * NC) : "memory", "scc"); } while (0)
        const int q = li >> 1, h = li & 1;
        const int f0 = ((q >> 2) & 1) | ((g & 1) << 1);
        const int rbase = (16 * g + q) * 64 + 8 * h;
#define WREAD(r) do { _Pragma("unroll") for (int ks = 0; ks < 2; ++ks) _Pragma("unroll") for (int t = 0; t < 4; ++t) { \
            v2i e[4]; \
            _Pragma("unroll") for (int u = 0; u < 4; ++u) \
                e[u] = __builtin_amdgcn_ds_read_tr8_b64_v2i32((LAS v2i*)(myl + 8192 + ((r) & 1) * 16384 + ks * 8192 + (u >> 1) * 4096 + (u & 1) * 512 + rbase + ((t ^ f0) * 16))); \
            wf[(r) * 2 + ks][t] = (v8i){e[0][0], e[0][1], e[1][0], e[1][1], e[2][0], e[2][1], e[3][0], e[3][1]}; } \
            asm volatile("s_waitcnt lgkmcnt(0)" ::: "memory"); } while (0)
        WLOAD(0); WLOAD(1);
        waitv<16>();
        if (WPF) {
#pragma unroll
            for (int k = 0; k < NPF; ++k) asm volatile("" :: "v"(pf[k]));
        }
        WREAD(0); WLOAD(2);
        waitv<16>(); WREAD(1); WLOAD(3);
        waitv<16>(); WREAD(2);
        asm volatile("s_load_dword %0, %2, 0x0\n\ts_load_dword %1, %3, 0x0" : "=&s"(sxv), "=&s"(swv) : "s"(a_sx), "s"(a_sw) : "memory");
        so_d = OWN_SO(0);
#pragma unroll
        for (int r = 0; r < 16; ++r) DMA(0, r);
        waitv<16>(); WREAD(3);
        asm volatile("" : "+s"(sxv), "+s"(swv));
        so_d = OWN_SO(1);
#pragma unroll
        for (int r = 0; r < 16; ++r) DMA(1, r);
        waitv<16>();
#pragma unroll
        for (int q2 = 0; q2 < 16; ++q2) XRD(0, q2);
    }
    TS(1);
    const float scale = sxv * swv;
    const unsigned call = ticket_get(tk);
    const char* const stg = me + OFF_STAGE + ((size_t)(call & 1) * NR + srcr) * XB;
    const char* const zslot = me + OFF_STAGE + ((size_t)(~call & 1) * NR + srcr) * XB;
    const __amdgpu_buffer_rsrc_t rs_z = mkrsrc(zslot + (cs * 4 + wid) * 16384, 16384);
    int vo_z = lane * 16;
    u32x4 zero = {0u, 0u, 0u, 0u};
    asm volatile("" : "+v"(zero));
#define ZERO() asm volatile("buffer_store_dwordx4 %1, %0, %2, 0 offen sc0 sc1\n\tv_add_u32 %0, 0x400, %0" : "+v"(vo_z) : "v"(zero), "s"(rs_z) : "memory")
#define ZDONE() asm volatile("s_mov_b64 exec, 1\n\tglobal_atomic_add_x2 %0, %1, %2 sc1\n\ts_mov_b64 exec, -1\n\ts_nop 1" :: "v"(0), "v"(1ull << 32), "s"(me + OFF_PCTR + (role - 1) * 128) : "memory")
    const __amdgpu_buffer_rsrc_t rs_out = mkrsrc(a_out, (unsigned)(NR * MS * NC * 4));
    int vo_x[4];
#pragma unroll
    for (int j = 0; j < 4; ++j) { vo_x[j] = lane * 16 + wid * 16384 + j * 4096; asm volatile("" : "+v"(vo_x[j])); }
    unsigned* const tmo = (unsigned*)(me + OFF_TMO);
    const char* const pollb = me + OFF_FLAGS + (srcr * NPW + wid * 4) * 128;
    const int pollv = lane * 8;
    v2i pv = {0, 0};
    v4f cin = {0.f, 0.f, 0.f, 0.f};
    const int wr_off = g * 256 + ((li ^ g) * 16);
    const int mrow = 4 * wid + g;
    const int rd_off = (li >> 2) * 1024 + (li & 3) * 256 + ((mrow ^ li) * 16);
    const int vo_out = mrow * (NC * 4) + cs * 256 + li * 16;
    int vo_st = (int)0x80000000u, vo_nx = dummy ? (int)0x80000000u : vo_out;
    v4f accA[4], accB[4], part[4], res = {0.f, 0.f, 0.f, 0.f}, ex;
#pragma unroll
    for (int t = 0; t < 4; ++t) accB[t] = (v4f){0.f, 0.f, 0.f, 0.f};

    const int oo_own = (int)a_rank * (int)XB * 2, oo_rem = (int)srcr * (int)XB * 2;
    int oo_p = 0, oo_c = oo_own + 2 * OWN_SO(0), oo_n = oo_own + 2 * OWN_SO(1);
    __amdgpu_buffer_rsrc_t rs_;
    int so_q, lim_n = NBLK; unsigned sp[2] = {0, 0}, force = 0, um, tm = 0;
    unsigned long long known = ~0ull << 32;
#define DECODE(pv_) do { const unsigned long long m0 = __builtin_amdgcn_ballot_w64((unsigned)pv_[0] >= call), m1 = __builtin_amdgcn_ballot_w64((unsigned)pv_[1] >= call); \
        unsigned a0 = (unsigned)m0 & (unsigned)(m0 >> 32), a1 = (unsigned)m1 & (unsigned)(m1 >> 32); a0 &= a0 >> 16; a1 &= a1 << 16; \
        known |= (a0 & 0xffffu) | (a1 & 0xffff0000u); asm volatile("" : "+s"(known)); } while (0)
#define POLL(pv_) asm volatile("global_load_dwordx2 %0, %1, %2 sc0 sc1" : "+v"(pv_) : "v"(pollv), "s"(pollb) : "memory")
#define DBGW0() do { } while (0)
#define DBGW() do { } while (0)
#define DBGR(c) do { } while (0)
#define LD(bf, q_) asm volatile("buffer_load_dwordx4 %0, %1, %2, %3 offen offset:%4 " X_BITS : "=v"(xb[bf][q_]) : "v"(vo_x[(q_) >> 2]), "s"(rs_), "s"(so_q), "i"(((q_) & 3) * 1024) : "memory")
#define PLAN(bf, n) do { int n_ = (n); asm volatile("" : "+s"(n_)); \
        asm volatile("s_bitcmp1_b64 %1, %2\n\ts_cselect_b32 %0, 0, 15\n\ts_or_b32 %0, %0, %3" : "=&s"(sp[bf]) : "s"(known), "s"(n_), "s"(force) : "scc"); \
        so_q = n_ * 65536; rs_ = mkrsrc(stg, n_ < lim_n ? XB : 0); asm volatile("" : "+s"(so_q)); } while (0)
    unsigned zm[4];
#define ZW(n) asm volatile("s_waitcnt vmcnt(%0)" :: "i"(n) : "memory")
#define XW(bf, q) "v"(xb[bf][q][0]), "v"(xb[bf][q][1]), "v"(xb[bf][q][2]), "v"(xb[bf][q][3])
#define ZM(bf, q) do { if (((q) & 3) == 0) asm volatile("v_min3_u32 %0, %1, %2, %3\n\tv_min_u32 %0, %0, %4" : "=&v"(zm[(q) >> 2]) : XW(bf, q)); \
        else asm volatile("v_min3_u32 %0, %1, %2, %0\n\tv_min3_u32 %0, %3, %4, %0" : "+v"(zm[(q) >> 2]) : XW(bf, q)); } while (0)
#define ZM2(bf, q) do { if (((q) & 3) == 0) asm volatile("v_min3_u32 %0, %1, %2, %3\n\tv_min3_u32 %0, %4, %5, %0\n\tv_min3_u32 %0, %6, %7, %0\n\tv_min_u32 %0, %0, %8" \
            : "=&v"(zm[(q) >> 2]) : XW(bf, q), XW(bf, (q) + 1)); \
        else asm volatile("v_min3_u32 %0, %1, %2, %0\n\tv_min3_u32 %0, %3, %4, %0\n\tv_min3_u32 %0, %5, %6, %0\n\tv_min3_u32 %0, %7, %8, %0" \
            : "+v"(zm[(q) >> 2]) : XW(bf, q), XW(bf, (q) + 1)); } while (0)
#define UT(u) do { if ((u) == 0) asm volatile("v_cmp_eq_u32 vcc, 0, %1\n\ts_cmp_lg_u64 vcc, 0\n\ts_addc_u32 %0, 2, 2" : "=s"(um) : "v"(zm[u]) : "vcc", "scc"); \
        else asm volatile("v_cmp_eq_u32 vcc, 0, %1\n\ts_cmp_lg_u64 vcc, 0\n\ts_addc_u32 %0, %0, %0" : "+s"(um) : "v"(zm[u]) : "vcc", "scc"); } while (0)
#define RELOAD(bf, c) do { so_q = (c) * 65536; _Pragma("unroll") for (int q = 0; q < 16; ++q) LD(bf, q); } while (0)
#define RETRY(bf, c) do { \
        DBGW0(); TS(43); \
        rs_ = mkrsrc(stg, XB); so_q = (c) * 65536; \
        unsigned t0 = 0, it = 0; \
        force = 0; lim_n = NBLK; \
        if (sp[bf]) { DBGR(c); if ((c) + 2 < NBLK) { force = 32; lim_n = 0; } } \
        while (sp[bf]) { \
            unsigned long long t1; \
            const unsigned keep = (known >> (c)) & 1 ? 0u : 0xfu; \
            _Pragma("unroll") for (int u = 0; u < 4; ++u) if (sp[bf] & (8u >> u)) { \
                _Pragma("unroll") for (int q = 0; q < 4; ++q) LD(bf, 4 * u + q); } \
            if ((++it & 3) == 0) { POLL(pv); asm volatile("global_load_dword %0, %1, %2 sc1" : "+v"(tm) : "v"(0), "s"(tmo) : "memory"); } \
            asm volatile("s_memrealtime %0\n\ts_waitcnt vmcnt(0) lgkmcnt(0)" : "=s"(t1) :: "memory"); \
            asm volatile("" : "+v"(pv), "+v"(tm)); \
            _Pragma("unroll") for (int u = 0; u < 4; ++u) { ZM2(bf, 4 * u); ZM2(bf, 4 * u + 2); UT(u); } \
            sp[bf] &= um & keep; \
            DECODE(pv); \
            if (it == 1) t0 = (unsigned)t1; \
            if (sp[bf] && ((unsigned)__builtin_amdgcn_readfirstlane(tm) == call || (unsigned)t1 - t0 > (unsigned)DEADLINE)) { \
                __hip_atomic_store(tmo, call, __ATOMIC_RELAXED, __HIP_MEMORY_SCOPE_AGENT); \
                known = ~0ull; sp[bf] = 0; cin = (v4f){__builtin_nanf(""), __builtin_nanf(""), __builtin_nanf(""), __builtin_nanf("")}; \
                asm volatile("s_nop 3" : "+v"(cin)); } \
        } \
        if (sp[(bf) ^ 1]) { sp[(bf) ^ 1] = (known >> ((c) + 1)) & 1 ? 0u : 0xfu; RELOAD((bf) ^ 1, (c) + 1); POLL(pv); } \
        asm volatile("" : "+s"(force), "+s"(lim_n), "+s"(known), "+s"(sp[0]), "+s"(sp[1])); \
        TS(44); DBGW(); } while (0)
#define RED_WRITE(acc, bf, t) *(LAS v4f*)((LAS char*)myl + (bf) * 4096 + (t) * 1024 + (wr_off ^ ((t) * 64))) = acc[t]
#define RED_BAR() asm volatile("s_waitcnt lgkmcnt(0)\n\ts_barrier" ::: "memory")
#define RED_READ(bf, w) part[w] = *(LAS v4f*)((LAS char*)lds + (w) * LW + (bf) * 4096 + rd_off)
#define ADD4(a, b) do { _Pragma("unroll") for (int r_ = 0; r_ < 4; ++r_) { a[r_] += b[r_]; asm volatile("" : "+v"(a[r_])); } } while (0)
#define EPI_A(r) do { res[r] = part[0][r] * scale; \
        ex[r] = __builtin_amdgcn_exp2f(__builtin_amdgcn_fmed3f(res[r], -60.f, 60.f) * -1.44269504088896f); asm volatile("" : "+v"(res[r]), "+v"(ex[r])); } while (0)
#define EPI_B(r) do { res[r] *= __builtin_amdgcn_rcpf(1.f + ex[r]); asm volatile("" : "+v"(res[r])); } while (0)
#define STORE_RES(vo, so) __builtin_amdgcn_raw_buffer_store_b128(__builtin_bit_cast(u32x4, res), rs_out, vo, so, OUT_AUX)
#define MF(acc, t, ks) do { if (ks == 0) asm volatile("v_mfma_f32_16x16x128_f8f6f4 %0, %1, %2, %3 cbsz:1" : "=&v"(acc[t]) : "a"(wf[ks][t]), "v"(xv), "v"(cin)); \
        else asm volatile("v_mfma_f32_16x16x128_f8f6f4 %0, %1, %2, %0 cbsz:1" : "+v"(acc[t]) : "a"(wf[ks][t]), "v"(xv)); } while (0)
#define FILL(s, bf, accp, vo) do { \
        if (!(DBG_X & 1)) { \
        if (s == 1 || s == 3 || s == 5 || s == 7) RED_WRITE(accp, (bf) ^ 1, s >> 1); \
        if (s == RS0) RED_BAR(); \
        if (s == RS0 || s == RS0 + 1) { RED_READ((bf) ^ 1, 2 * (s - RS0)); RED_READ((bf) ^ 1, 2 * (s - RS0) + 1); } \
        if (s == RS0 + 4) ADD4(part[0], part[1]); \
        if (s == RS0 + 5) ADD4(part[2], part[3]); \
        if (s == RS0 + 6) ADD4(part[0], part[2]); } \
        if (!(DBG_X & 2)) { \
        if (s >= RS0 + 7 && s < RS0 + 11) EPI_A(s - RS0 - 7); \
        if (s >= RS0 + 11 && s < RS0 + 15) EPI_B(s - RS0 - 11); } \
        if (s == RS0 + 16 && !(DBG_X & 4)) STORE_RES(vo, oo_p); \
        } while (0)
#define XV(bf, ks) const u32x4 lo_ = xb[bf][2 * ks], hi_ = xb[bf][2 * ks + 1]; \
        const v8i xv = {(int)lo_[0], (int)lo_[1], (int)lo_[2], (int)lo_[3], (int)hi_[0], (int)hi_[1], (int)hi_[2], (int)hi_[3]}
#define OSTEP(j, bf, acc, accp) do { \
        so_d = OWN_SO((j) + 2); if ((bf) == 1) rs_own = mkrsrc(a_x, (j) < 3 ? XB : 0); \
        asm volatile("s_waitcnt lgkmcnt(0)" ::: "memory"); SB(); \
        _Pragma("unroll") for (int ks = 0; ks < 8; ++ks) { XV(bf, ks); \
            _Pragma("unroll") for (int t = 0; t < 4; ++t) { const int sl = 4 * ks + t; \
                MF(acc, t, ks); SB(); \
                if (sl == 14) waitv<15>(); \
                if (sl < 16) DMA(bf, sl); \
                if (sl >= 14 && sl < 30) XRD((bf) ^ 1, sl - 14); \
                FILL(sl, bf, accp, vo_st); \
                if (sl == 30) { vo_st = vo_nx; vo_nx = vo_out; asm volatile("" : "+v"(vo_st), "+v"(vo_nx)); } \
                if (sl == 1 && (bf) == 1) PLAN(1, 0); \
                if (t == 3 && (bf) == 1 && (j) == 3) { LD(1, 2 * ks); LD(1, 2 * ks + 1); } \
                SB(); } } \
        oo_p = oo_c; oo_c = oo_n; oo_n = oo_own + 2 * so_d; asm volatile("" : "+s"(oo_p), "+s"(oo_c), "+s"(oo_n)); \
        if (DBG_ALLSTEPS) TS(2 + (j)); \
        } while (0)
#define PK(op, d, a, b) asm volatile(op " %0, %1, %2" : "=v"(d) : "v"(a), "v"(b))
#define PKS(op, d, a, b) asm volatile(op " %0, %1, %2 op_sel_hi:[1,0]" : "=v"(d) : "v"(a), "s"(b))
#define LO(v) __builtin_shufflevector(v, v, 0, 1)
#define HI(v) __builtin_shufflevector(v, v, 2, 3)
#define RD2(bf, w) part[w] = *(LAS v4f*)((LAS char*)((w) < 2 ? rd_a : rd_b) + ((w) & 1) * LW + (bf) * 4096)
#define CLAMP(v, i) do { float f_; asm volatile("v_med3_f32 %0, %1, %2, %3" : "=v"(f_) : "v"(v[i]), "s"(-86.5617024533f), "v"(lim)); v[i] = f_; } while (0)
#define TR(op, v, i) do { float f_; asm volatile(op " %0, %1" : "=v"(f_) : "v"(v[i])); v[i] = f_; } while (0)
#define GSTEP(n, bf, acc, accp) do { \
        if (__builtin_expect((sp[bf] | force) != 0, 0)) RETRY(bf, (n) - 2); \
        SB(); \
        _Pragma("unroll") for (int ks = 0; ks < 8; ++ks) { \
            XV(bf, ks); \
            _Pragma("unroll") for (int t = 0; t < 4; ++t) { const int sl = 4 * ks + t; constexpr int b = (bf) ^ 1; \
                MF(acc, t, ks); SB(); \
                if (sl == 0) STORE_RES(vo_out, o2);        \
                if (sl == 1) PLAN(bf, n); \
                if (sl == 2) RED_WRITE(accp, b, 0); \
                if (sl == 2 && (bf) == 1 && (n) == zn) ZDONE(); \
                if (sl == 3) { o2 = o1; o1 = o0; o0 = 2 * so_q + oo_r1; asm volatile("" : "+s"(o2), "+s"(o1), "+s"(o0)); } \
                if (sl >= 4 && sl <= 6) RED_WRITE(accp, b, sl - 3); \
                  \
                if (sl == 7) { ZW(18); ZM2(b, 0); } \
                if (sl == 8) { RED_BAR(); RD2(b, 0); RD2(b, 1); RD2(b, 2); RD2(b, 3); } \
                if (sl == 9) { ZW(18); ZM2(b, 2); } \
                if (sl == 10) UT(0); \
                if (sl == 11) { PK("v_pk_add_f32", qa, LO(part[0]), LO(part[1])); PK("v_pk_add_f32", qb, HI(part[0]), HI(part[1])); } \
                if (sl == 12) { PK("v_pk_add_f32", qc, LO(part[2]), LO(part[3])); PK("v_pk_add_f32", qd, HI(part[2]), HI(part[3])); \
                                PK("v_pk_add_f32", qa, qa, qc); PK("v_pk_add_f32", qb, qb, qd); } \
                if (sl == 13) { ZW(18); ZM2(b, 4); PKS("v_pk_mul_f32", qc, qa, nsl); PKS("v_pk_mul_f32", qd, qb, nsl); } \
                if (sl == 14) { PKS("v_pk_mul_f32", qa, qa, scl); PKS("v_pk_mul_f32", qb, qb, scl); CLAMP(qc, 0); CLAMP(qc, 1); CLAMP(qd, 0); CLAMP(qd, 1); } \
                if (sl == 15) TR("v_exp_f32", qc, 0); \
                if (sl == 16) { TR("v_exp_f32", qc, 1); asm volatile("s_nop 0"); PKS("v_pk_add_f32", qc, qc, one); } \
                if (sl == 17) { TR("v_exp_f32", qd, 0); ZW(18); ZM(b, 6); } \
                if (sl == 18) { TR("v_exp_f32", qd, 1); ZM(b, 7); } \
                if (sl == 19) TR("v_rcp_f32", qc, 0); \
                if (sl == 20) { TR("v_rcp_f32", qc, 1); PKS("v_pk_add_f32", qd, qd, one); } \
                if (sl == 21) { TR("v_rcp_f32", qd, 0); ZW(18); ZM(b, 8); } \
                if (sl == 22) { TR("v_rcp_f32", qd, 1); ZM(b, 9); } \
                if (sl == 23) { UT(1); PK("v_pk_mul_f32", qa, qa, qc); PK("v_pk_mul_f32", qb, qb, qd); res = __builtin_shufflevector(qa, qb, 0, 1, 2, 3); } \
                if (sl == 24) ZERO(); \
                if (sl == 25) { ZW(18); ZM2(b, 10); } \
                if (sl == 26) UT(2); \
                if (sl == 28) { ZW(18); ZM2(b, 12); } \
                if (sl == 29) { ZW(16); ZM2(b, 14); } \
                if (sl == 30) { UT(3); sp[b] &= um; asm volatile("" : "+s"(sp[b])); } \
                if (t == 3) { LD(bf, 2 * ks); LD(bf, 2 * ks + 1); } \
                SB(); } } \
        if (DBG_ALLSTEPS) TS(5 + (n)); else if ((n) == 16) TS(21); \
        } while (0)

    POLL(pv);
    for (int j = 0; j < 4; j += 2) { OSTEP(j, 0, accA, accB); OSTEP(j + 1, 1, accB, accA); }
    if (!DBG_ALLSTEPS) TS(2);
    int o2 = oo_own + 2 * OWN_SO(2), o1 = oo_p, o0 = oo_c;
    const int oo_r1 = oo_rem - 131072;
    v2f qa, qb, qc, qd;
    const unsigned long long scl = (unsigned)__builtin_amdgcn_readfirstlane(__builtin_bit_cast(int, scale)),
        nsl = (unsigned)__builtin_amdgcn_readfirstlane(__builtin_bit_cast(int, scale * -1.44269504088896f)), one = 0x3f800000u;
    float lim = 86.5617024533f;
    LAS char* rd_a = (LAS char*)lds + rd_off; LAS char* rd_b = rd_a + 2 * LW;
    asm volatile("" : "+v"(lim), "+v"(rd_a), "+v"(rd_b));
    int zn = wid == 0 ? 20 : -64;
    asm volatile("" : "+s"(zn));
    __builtin_amdgcn_s_waitcnt(0xc07f);
    for (int n = 0; n <= NBLK; n += 2) { if (n > 0) GSTEP(n, 1, accB, accA); GSTEP(n + 1, 0, accA, accB); }
    if (!DBG_ALLSTEPS) TS(39);
    STORE_RES(vo_out, o2);
    asm volatile("s_nop 15" : "+v"(accA[0]), "+v"(accA[1]), "+v"(accA[2]), "+v"(accA[3]));
    RED_WRITE(accA, 0, 0); RED_WRITE(accA, 0, 1); RED_WRITE(accA, 0, 2); RED_WRITE(accA, 0, 3);
    RED_BAR(); RED_READ(0, 0); RED_READ(0, 1); RED_READ(0, 2); RED_READ(0, 3);
    ADD4(part[0], part[1]); ADD4(part[2], part[3]); ADD4(part[0], part[2]);
    EPI_A(0); EPI_A(1); EPI_A(2); EPI_A(3); EPI_B(0); EPI_B(1); EPI_B(2); EPI_B(3);
    STORE_RES(vo_out, o1);
    SB();
    asm volatile("s_waitcnt vmcnt(0)" : "+v"(pv) :: "memory");
#pragma unroll
    for (int q = 0; q < 16; ++q) asm volatile("" :: "v"(xb[0][q]), "v"(xb[1][q]));
}

extern "C" void kernel_launch(int rank, int nranks, ncclComm_t comm, hipStream_t stream,
                              void* const* d_in, const int* in_sizes, int n_in,
                              void* d_out, int out_size, const int* in_dtypes, int out_dtype,
                              void* const* d_out_all, void* const* d_ws_all, size_t ws_bytes) {
    (void)comm; (void)in_dtypes; (void)out_dtype; (void)d_out_all;
    if (nranks != NR || rank < 0 || rank >= NR || n_in < 4 || ws_bytes < WS_NEED || in_sizes[0] != (int)XB || in_sizes[1] != KD * NC
        || out_size != NR * MS * NC) { fprintf(stderr, "unexpected problem shape\n"); return; }
    Params p;
    for (int i = 0; i < NR; ++i) p.dst[i] = (char*)d_ws_all[(rank + 1 + i) % NR];
    p.x = (const char*)d_in[0]; p.w = (const char*)d_in[1]; p.sx = (const float*)d_in[2]; p.sw = (const float*)d_in[3];
    p.out = (float*)d_out; p.rank = (unsigned)rank; p.pad = 0;
    ag_gemm_silu<<<GRID, 256, 0, stream>>>(p);
    HIP_CHECK(hipGetLastError());
}
```

```cpp
#include <hip/hip_runtime.h>
#include <rccl/rccl.h>
#include <cstdio>
#include <cstdint>

#define HIP_CHECK(expr) do { hipError_t _e = (expr); if (_e != hipSuccess) { \
    fprintf(stderr, "HIP error %s at %s:%d\n", hipGetErrorString(_e), __FILE__, __LINE__); \
    return; } } while (0)

typedef unsigned u32x4 __attribute__((ext_vector_type(4)));
typedef int v2i __attribute__((ext_vector_type(2)));
typedef int v4i __attribute__((ext_vector_type(4)));
typedef int v8i __attribute__((ext_vector_type(8)));
typedef float v4f __attribute__((ext_vector_type(4)));
typedef float v2f __attribute__((ext_vector_type(2)));
#define LAS __attribute__((address_space(3)))
#define GAS __attribute__((address_space(1)))
#define CAS __attribute__((address_space(4)))
#define SB() __builtin_amdgcn_sched_barrier(0)

#define OUT_AUX 17
#define X_BITS "sc0 sc1"
#define WPF 1
#define SLEEP 1
#define RS0 10
#define DBG_ALLSTEPS 0
#define DBG_X 0
#define MAPR 8
#define COAL 1
#define XSTEP 65536
#define EXTRA 1
#define PFB 0
#define NOCONS 0
#define CDELAY 0
#define ZS0 1
#define XG 2

constexpr int NR = 8;
constexpr int MS = 512, KD = 4096, NC = 2048;
constexpr unsigned XB = MS * KD;
constexpr int NBLK = MS / 16;
constexpr int NPW = 16;
constexpr int GRID = 256;
constexpr int LW = 40960;
constexpr size_t OFF_CTR = 0;
constexpr size_t OFF_TMO = 1024 * 128;
constexpr size_t OFF_PCTR = OFF_TMO + 4096;
constexpr size_t OFF_FLAGS = OFF_PCTR + 4096;
constexpr unsigned PER = (NBLK + 1 - EXTRA) * NPW;
constexpr unsigned GATE = (NBLK - 2) * NPW;
constexpr unsigned NZW = 32;
constexpr size_t OFF_DBG = 256 << 10;
constexpr size_t OFF_STAGE = 1 << 20;
constexpr size_t WS_NEED = OFF_STAGE + 2 * (size_t)NR * XB;
constexpr unsigned long long DEADLINE = 40000000ull;

struct Params {
    char* dst[NR];
    const char* x; const char* w; const float* sx; const float* sw; float* out;
    unsigned rank; unsigned pad;
};

template <int N> __device__ __forceinline__ void waitv() { asm volatile("s_waitcnt vmcnt(%0)" :: "i"(N) : "memory"); }

__device__ __forceinline__ __amdgpu_buffer_rsrc_t mkrsrc(const void* p, unsigned bytes) {
    return __builtin_amdgcn_make_buffer_rsrc((void*)p, (short)0, (int)bytes, 0x00020000);
}

__device__ __forceinline__ unsigned ticket_issue(char* ctr) {
    unsigned t; unsigned long long sv;
    asm volatile("s_mov_b64 %1, exec\n\ts_mov_b64 exec, 1\n\tglobal_atomic_add %0, %2, %3, %4 sc0\n\ts_mov_b64 exec, %1"
                 : "=&v"(t), "=&s"(sv) : "v"(0), "v"(1), "s"(ctr) : "memory");
    return t;
}
__device__ __forceinline__ unsigned ticket_get(unsigned t) {
    asm volatile("" : "+v"(t));
    return __builtin_amdgcn_readfirstlane(t) + 1;
}

#define TS(slot) do { } while (0)

__global__ __launch_bounds__(256, 1) void ag_gemm_silu(Params p) {
    __shared__ __attribute__((aligned(16))) char lds[4 * LW];
    const int lane = threadIdx.x & 63;
    const int wid = __builtin_amdgcn_readfirstlane(threadIdx.x >> 6);
    const int bid = blockIdx.x;
    const int gwave = bid * 4 + wid;
    char* const me = p.dst[NR - 1];
    TS(0);
    constexpr int SL = 4 * XG;
    const int xw = (bid >> 3) - 4, gi = (bid & (XG - 1)) * 28 + xw;
    const int role = bid < 32 ? 0 : 1 + gi / SL, cs = ((bid & 7) / XG) * SL + gi % SL, pidx = bid;

    if (__builtin_expect(role == 0, 1)) {
        const int gw = pidx * 4 + wid;
        if (gw >= 7 * NPW) {
            const char* sb = p.x + (gw - 7 * NPW) * 4096;
            u32x4 t = {0u, 0u, 0u, 0u};
#pragma unroll
            for (int b = 1; b <= PFB; ++b) {
                sb += 65536;
#pragma unroll
                for (int j = 0; j < 4; ++j) asm volatile("global_load_dwordx4 %0, %1, %2 offset:%3" : "=v"(t) : "v"(lane * 16), "s"(sb), "i"(j * 1024) : "memory");
            }
            asm volatile("s_waitcnt vmcnt(0)" : "+v"(t) :: "memory");
            TS(48);
            return;
        }
        const int pi = gw % 7;
        const int vsrc = (lane & 15) * KD + (lane >> 4) * 16, vdst = lane * 16;
        u32x4 buf[4];
#define LOADB(i) asm volatile("global_load_dwordx4 %0, %1, %2 offset:%3" : "=v"(buf[i]) : "v"(vsrc), "s"(sb), "i"((i) * 64) : "memory")
#define STOREB(i) asm volatile("global_store_dwordx4 %0, %1, %2 offset:%3 sc0 sc1\n\ts_nop 1" :: "v"(vdst), "v"(buf[i]), "s"(db), "i"((i) * 1024) : "memory")
#define WAITB(n, i) asm volatile("s_waitcnt vmcnt(%2)" : "+v"(buf[i]), "+v"(tk) : "i"(n) : "memory")
#define DRAW(inc) asm volatile("s_mov_b64 exec, 1\n\tglobal_atomic_add_x2 %0, %1, %2, %3 sc0 sc1\n\ts_mov_b64 exec, -1" : "=&v"(tk) : "v"(0), "v"((unsigned long long)(inc)), "s"(ctr) : "memory")
#define FLAG() asm volatile("s_mov_b64 exec, 1\n\tglobal_store_dword %0, %1, %2 sc0 sc1\n\ts_mov_b64 exec, -1" :: "v"(0), "v"(call), "s"(fa) : "memory")
        const char* px = p.x; char* pw = p.dst[pi]; char* pm = me; unsigned rk = p.rank;
        asm volatile("" : "+s"(px), "+s"(pw), "+s"(pm), "+s"(rk));
        char* const ctr = pm + OFF_PCTR + pi * 128;
        unsigned u = gw / 7;
        u32x4 ext[EXTRA + 1][4];
        v2i tk;
        const char* sb = px + u * 256;
#pragma unroll
        for (int e = 0; e < EXTRA; ++e, sb += XSTEP) {
#pragma unroll
            for (int i = 0; i < 4; ++i) asm volatile("global_load_dwordx4 %0, %1, %2 offset:%3" : "=v"(ext[e][i]) : "v"(COAL ? (lane >> 2) * KD + (lane & 3) * 16 : vsrc), "s"(sb), "i"(i * 64) : "memory");
            if (e == 0) DRAW(1);
        }
        LOADB(0); LOADB(1); LOADB(2); LOADB(3);
        if (!EXTRA) DRAW(1);
#define FIXUP(e) do { asm volatile("" : "+v"(ext[e][0]), "+v"(ext[e][1]), "+v"(ext[e][2]), "+v"(ext[e][3])); \
        if (COAL) { _Pragma("unroll") for (int i = 0; i < 16; ++i) \
            ext[e][i >> 2][i & 3] = (unsigned)__builtin_amdgcn_ds_bpermute(((lane & 15) * 4 + (lane >> 4)) * 4, (int)ext[e][i >> 2][i & 3]); \
            asm volatile("" : "+v"(ext[e][0]), "+v"(ext[e][1]), "+v"(ext[e][2]), "+v"(ext[e][3])); } } while (0)
        if (EXTRA) { waitv<EXTRA * 4 + 1>(); FIXUP(0); }
        WAITB(EXTRA * 4, 0);
        const unsigned call = (unsigned)__builtin_amdgcn_readfirstlane(tk[0]) / PER + 1;
        const unsigned tbase = (call - 1) * PER - (EXTRA + 1) * NPW;
        char* const stage = pw + OFF_STAGE + ((size_t)(call & 1) * NR + rk) * XB;
        char* const flags = pw + OFF_FLAGS + rk * NPW * 128;
        char* fa = EXTRA ? flags + u * 128 : pm + OFF_TMO + 2048;
        int more = EXTRA - 1;
#pragma unroll
        for (int e = 0; e < EXTRA; ++e, u += NPW) {
            if (e) { waitv<EXTRA * 4>(); FIXUP(e); }
#pragma unroll
            for (int i = 0; i < 4; ++i)
                asm volatile("global_store_dwordx4 %0, %1, %2 offset:%3 sc0 sc1\n\ts_nop 1" :: "v"(vdst), "v"(ext[e][i]), "s"(stage + u * 4096), "i"(i * 1024) : "memory");
        }
        WAITB(EXTRA * 4, 0);
        TS(1);
        while (true) {
            WAITB(7, 0);
            const unsigned un = __builtin_amdgcn_readfirstlane(tk[0]) - tbase, ul = un < NBLK * NPW ? un : 0;
            if (__builtin_expect(!NOCONS && un >= GATE && (unsigned)__builtin_amdgcn_readfirstlane(tk[1]) < call * NZW, 0)) {
                const unsigned long long t0 = __builtin_amdgcn_s_memrealtime();
                do {
                    if (__builtin_amdgcn_s_memrealtime() - t0 > DEADLINE) {
                        __hip_atomic_store((unsigned*)(pm + OFF_TMO), call, __ATOMIC_RELAXED, __HIP_MEMORY_SCOPE_AGENT); return; }
                    __builtin_amdgcn_s_sleep(8);
                    DRAW(0); WAITB(0, 0);
                } while ((unsigned)__builtin_amdgcn_readfirstlane(tk[1]) < call * NZW);
            }
            DRAW(1);
            char* const db = stage + u * 4096;
            sb = px + (ul >> 4) * 65536 + (ul & 15) * 256;
            STOREB(0); LOADB(0);
            WAITB(8, 1); STOREB(1); LOADB(1);
            WAITB(8, 2); STOREB(2); LOADB(2);
            WAITB(7, 3);
            FLAG();
            if (EXTRA > 1) for (; more > 0; --more) { fa += 8; FLAG(); }
            STOREB(3); LOADB(3);
            fa = flags + (u & 15) * 128 + ((u >> 4) & 15) * 8 + (u >> 8) * 4;
            if (DBG_ALLSTEPS) TS(3 + (u >> 4));
            u = un;
            if (u >= NBLK * NPW) break;
        }
        WAITB(0, 0);
        FLAG();
        asm volatile("" :: "v"(buf[1]), "v"(buf[2]), "v"(buf[3]));
        TS(2);
        return;
    }

    if (NOCONS) { if (p.sx[0] != 123.f) return; }
    if (CDELAY) { const unsigned long long t0 = __builtin_amdgcn_s_memrealtime(); while (__builtin_amdgcn_s_memrealtime() - t0 < CDELAY) __builtin_amdgcn_s_sleep(2); }
    const char* a_x = p.x; const char* a_w = p.w; const float* a_sx = p.sx; const float* a_sw = p.sw; float* a_out = p.out; unsigned a_rank = p.rank;
    asm volatile("" : "+s"(a_x), "+s"(a_w), "+s"(a_sx), "+s"(a_sw), "+s"(a_out), "+s"(a_rank));
    const unsigned srcr = (a_rank + role) % NR;
    const int g = lane >> 4, li = lane & 15, sub = cs & 3;
    char* const myl = lds + wid * LW;
    const unsigned lbase = (unsigned)(size_t)(LAS char*)myl;
    const unsigned tk = ticket_issue(me + OFF_CTR + (size_t)gwave * 128);
    const __amdgpu_buffer_rsrc_t rs_w = mkrsrc(a_w, (unsigned)(KD * NC));
    constexpr int NPF = (SL * 2048 + 112 * 64 - 1) / (112 * 64);
    unsigned pf[NPF];
    if (WPF) {
#pragma unroll
        for (int k = 0; k < NPF; ++k) {
            const int l = ((k * 28 + xw) * 4 + wid) * 64 + lane;
            asm volatile("buffer_load_dword %0, %1, %2, 0 offen" : "=v"(pf[k]) : "v"((l / (SL / 2)) * NC + (cs / SL) * SL * 64 + (l % (SL / 2)) * 128), "s"(rs_w) : "memory");
        }
    }
    float sxv, swv;
    u32x4 xb[2][16];
    v8i wf[8][4];
    const bool dummy = role >= 5;
#define OWN_SO(j) (((j) == 0 ? (dummy ? NBLK - 1 : 27 + role) : 7 * (((j) - 1 + sub) & 3) + role - 1) * 65536)
    int vrow[16];
#pragma unroll
    for (int r = 0; r < 16; ++r) vrow[r] = r * KD + wid * 1024 + ((lane ^ r) * 16);
    __amdgpu_buffer_rsrc_t rs_own = mkrsrc(a_x, XB);
    int so_d;
#define DMA(bf, r) asm volatile("s_add_u32 m0, %0, %1\n\ts_nop 0\n\tbuffer_load_dwordx4 %2, %3, %4 offen lds" \
        :: "s"(lbase), "i"(8192 + (bf) * 16384 + (r) * 1024), "v"(vrow[r]), "s"(rs_own), "s"(so_d) : "memory", "scc")
    int xa[4];
#pragma unroll
    for (int j = 0; j < 4; ++j) xa[j] = li * 1024 + (((4 * j + g) ^ li) * 16);
#define XRD(bf, q_) xb[bf][q_] = *(LAS u32x4*)((LAS char*)myl + 8192 + (bf) * 16384 + ((q_) >> 2) * 256 + xa[(q_) & 3])

    {
        const int kr = lane >> 2;
        const int wvA = (wid * 1024 + kr) * NC + cs * 64 + (((lane & 3) ^ ((kr >> 2) & 1)) * 16);
        const int wvB = wvA ^ 32;
#define WLOAD(r) do { _Pragma("unroll") for (int j = 0; j < 16; ++j) \
            asm volatile("s_add_u32 m0, %0, %1\n\ts_nop 0\n\tbuffer_load_dwordx4 %2, %3, %4 offen lds" \
                :: "s"(lbase), "i"(8192 + ((r) & 1) * 16384 + j * 1024), "v"((j & 1) ? wvB : wvA), "s"(rs_w), "s"(((r) * 256 + j * 16) * NC) : "memory", "scc"); } while (0)
        const int q = li >> 1, h = li & 1;
        const int f0 = ((q >> 2) & 1) | ((g & 1) << 1);
        const int rbase = (16 * g + q) * 64 + 8 * h;
#define WREAD(r) do { _Pragma("unroll") for (int ks = 0; ks < 2; ++ks) _Pragma("unroll") for (int t = 0; t < 4; ++t) { \
            v2i e[4]; \
            _Pragma("unroll") for (int u = 0; u < 4; ++u) \
                e[u] = __builtin_amdgcn_ds_read_tr8_b64_v2i32((LAS v2i*)(myl + 8192 + ((r) & 1) * 16384 + ks * 8192 + (u >> 1) * 4096 + (u & 1) * 512 + rbase + ((t ^ f0) * 16))); \
            wf[(r) * 2 + ks][t] = (v8i){e[0][0], e[0][1], e[1][0], e[1][1], e[2][0], e[2][1], e[3][0], e[3][1]}; } \
            asm volatile("s_waitcnt lgkmcnt(0)" ::: "memory"); } while (0)
        WLOAD(0); WLOAD(1);
        waitv<16>();
        if (WPF) {
#pragma unroll
            for (int k = 0; k < NPF; ++k) asm volatile("" :: "v"(pf[k]));
        }
        WREAD(0); WLOAD(2);
        waitv<16>(); WREAD(1); WLOAD(3);
        waitv<16>(); WREAD(2);
        asm volatile("s_load_dword %0, %2, 0x0\n\ts_load_dword %1, %3, 0x0" : "=&s"(sxv), "=&s"(swv) : "s"(a_sx), "s"(a_sw) : "memory");
        so_d = OWN_SO(0);
#pragma unroll
        for (int r = 0; r < 16; ++r) DMA(0, r);
        waitv<16>(); WREAD(3);
        asm volatile("" : "+s"(sxv), "+s"(swv));
        so_d = OWN_SO(1);
#pragma unroll
        for (int r = 0; r < 16; ++r) DMA(1, r);
        waitv<16>();
#pragma unroll
        for (int q2 = 0; q2 < 16; ++q2) XRD(0, q2);
    }
    TS(1);
    const float scale = sxv * swv;
    const unsigned call = ticket_get(tk);
    const char* const stg = me + OFF_STAGE + ((size_t)(call & 1) * NR + srcr) * XB;
    const char* const zslot = me + OFF_STAGE + ((size_t)(~call & 1) * NR + srcr) * XB;
    const __amdgpu_buffer_rsrc_t rs_z = mkrsrc(zslot + (cs * 4 + wid) * 16384, 16384);
    int vo_z = lane * 16;
    u32x4 zero = {0u, 0u, 0u, 0u};
    asm volatile("" : "+v"(zero));
#define ZERO() asm volatile("buffer_store_dwordx4 %1, %0, %2, 0 offen sc0 sc1\n\tv_add_u32 %0, 0x400, %0" : "+v"(vo_z) : "v"(zero), "s"(rs_z) : "memory")
#define ZDONE() asm volatile("s_mov_b64 exec, 1\n\tglobal_atomic_add_x2 %0, %1, %2 sc1\n\ts_mov_b64 exec, -1\n\ts_nop 1" :: "v"(0), "v"(1ull << 32), "s"(me + OFF_PCTR + (role - 1) * 128) : "memory")
    const __amdgpu_buffer_rsrc_t rs_out = mkrsrc(a_out, (unsigned)(NR * MS * NC * 4));
    int vo_x[4];
#pragma unroll
    for (int j = 0; j < 4; ++j) { vo_x[j] = lane * 16 + wid * 16384 + j * 4096; asm volatile("" : "+v"(vo_x[j])); }
    unsigned* const tmo = (unsigned*)(me + OFF_TMO);
    const char* const pollb = me + OFF_FLAGS + (srcr * NPW + wid * 4) * 128;
    const int pollv = lane * 8;
    v2i pv = {0, 0};
    v4f cin = {0.f, 0.f, 0.f, 0.f};
    const int wr_off = g * 256 + ((li ^ g) * 16);
    const int mrow = 4 * wid + g;
    const int rd_off = (li >> 2) * 1024 + (li & 3) * 256 + ((mrow ^ li) * 16);
    const int vo_out = mrow * (NC * 4) + cs * 256 + li * 16;
    int vo_st = (int)0x80000000u, vo_nx = dummy ? (int)0x80000000u : vo_out;
    v4f accA[4], accB[4], part[4], res = {0.f, 0.f, 0.f, 0.f}, ex;
#pragma unroll
    for (int t = 0; t < 4; ++t) accB[t] = (v4f){0.f, 0.f, 0.f, 0.f};

    const int oo_own = (int)a_rank * (int)XB * 2, oo_rem = (int)srcr * (int)XB * 2;
    int oo_p = 0, oo_c = oo_own + 2 * OWN_SO(0), oo_n = oo_own + 2 * OWN_SO(1);
    __amdgpu_buffer_rsrc_t rs_;
    int so_q; unsigned sp[2] = {0, 0};
    unsigned known = 0;
#define DECODE(pv_) do { const unsigned long long m0 = __builtin_amdgcn_ballot_w64((unsigned)pv_[0] >= call), m1 = __builtin_amdgcn_ballot_w64((unsigned)pv_[1] >= call); \
        unsigned a0 = (unsigned)m0 & (unsigned)(m0 >> 32), a1 = (unsigned)m1 & (unsigned)(m1 >> 32); a0 &= a0 >> 16; a1 &= a1 << 16; \
        known |= (a0 & 0xffffu) | (a1 & 0xffff0000u); asm volatile("" : "+s"(known)); } while (0)
#define POLL(pv_) asm volatile("global_load_dwordx2 %0, %1, %2 sc0 sc1" : "+v"(pv_) : "v"(pollv), "s"(pollb) : "memory")
#define DBGW0() do { } while (0)
#define DBGW() do { } while (0)
#define DBGR() do { } while (0)
#define LD(bf, q_) asm volatile("buffer_load_dwordx4 %0, %1, %2, %3 offen offset:%4 " X_BITS : "=v"(xb[bf][q_]) : "v"(vo_x[(q_) >> 2]), "s"(rs_), "s"(so_q), "i"(((q_) & 3) * 1024) : "memory")
#define PLAN(bf, n) do { int n_ = (n); asm volatile("" : "+s"(n_)); \
        sp[bf] = n_ < NBLK ? (1u << n_) & ~known : 0u; so_q = n_ * 65536; \
        rs_ = mkrsrc(stg, n_ < NBLK ? XB : 0); asm volatile("" : "+s"(sp[bf]), "+s"(so_q)); } while (0)
    unsigned zm0, zm1;
#define ZW(n) asm volatile("s_waitcnt vmcnt(%0)" :: "i"(n) : "memory")
#define ZM(bf, q) do { if ((q) == 0) asm volatile("v_min_u32 %0, %2, %3\n\tv_min_u32 %1, %4, %5" : "=&v"(zm0), "=&v"(zm1) \
            : "v"(xb[bf][q][0]), "v"(xb[bf][q][1]), "v"(xb[bf][q][2]), "v"(xb[bf][q][3])); \
        else asm volatile("v_min3_u32 %0, %2, %3, %0\n\tv_min3_u32 %1, %4, %5, %1" : "+v"(zm0), "+v"(zm1) \
            : "v"(xb[bf][q][0]), "v"(xb[bf][q][1]), "v"(xb[bf][q][2]), "v"(xb[bf][q][3])); } while (0)
#define ZM2(bf, q) do { if ((q) == 0) { ZM(bf, 0); ZM(bf, 1); } else asm volatile( \
            "v_min3_u32 %0, %2, %3, %0\n\tv_min3_u32 %1, %4, %5, %1\n\tv_min3_u32 %0, %6, %7, %0\n\tv_min3_u32 %1, %8, %9, %1" : "+v"(zm0), "+v"(zm1) \
            : "v"(xb[bf][q][0]), "v"(xb[bf][q][1]), "v"(xb[bf][q][2]), "v"(xb[bf][q][3]), \
              "v"(xb[bf][(q) + 1][0]), "v"(xb[bf][(q) + 1][1]), "v"(xb[bf][(q) + 1][2]), "v"(xb[bf][(q) + 1][3])); } while (0)
#define ZTEST(sp) do { if (__builtin_amdgcn_ballot_w64(min(zm0, zm1) == 0) == 0) sp = 0; asm volatile("" : "+s"(sp)); } while (0)
#define RELOAD(bf, c) do { so_q = (c) * 65536; _Pragma("unroll") for (int q = 0; q < 16; ++q) LD(bf, q); } while (0)
#define RETRY(bf, c) do { \
        unsigned long long t0 = 0; \
        DBGW0(); TS(43); DBGR(); \
        asm volatile("" : "+v"(pv)); DECODE(pv); \
        rs_ = mkrsrc(stg, XB); \
        do { \
            unsigned tm; \
            sp[bf] &= ~known; \
            RELOAD(bf, c); \
            POLL(pv); \
            asm volatile("global_load_dword %1, %2, %3 sc1\n\ts_waitcnt vmcnt(0)" : "+v"(pv), "=&v"(tm) : "v"(0), "s"(tmo) : "memory"); \
            DECODE(pv); \
            _Pragma("unroll") for (int q = 0; q < 16; q += 2) ZM2(bf, q); \
            ZTEST(sp[bf]); \
            const unsigned long long t1 = __builtin_amdgcn_s_memrealtime(); if (t0 == 0) t0 = t1; \
            if (sp[bf] && ((unsigned)__builtin_amdgcn_readfirstlane(tm) == call || t1 - t0 > DEADLINE)) { \
                __hip_atomic_store(tmo, call, __ATOMIC_RELAXED, __HIP_MEMORY_SCOPE_AGENT); \
                known = ~0u; sp[bf] = 0; cin = (v4f){__builtin_nanf(""), __builtin_nanf(""), __builtin_nanf(""), __builtin_nanf("")}; \
                asm volatile("s_nop 3" : "+v"(cin)); } \
            if (SLEEP) __builtin_amdgcn_s_sleep(SLEEP); \
        } while (sp[bf]); \
        if (sp[(bf) ^ 1]) { sp[(bf) ^ 1] &= ~known; RELOAD((bf) ^ 1, (c) + 1); POLL(pv); POLL(pv); } \
        TS(44); DBGW(); } while (0)
#define RED_WRITE(acc, bf, t) *(LAS v4f*)((LAS char*)myl + (bf) * 4096 + (t) * 1024 + (wr_off ^ ((t) * 64))) = acc[t]
#define RED_BAR() asm volatile("s_waitcnt lgkmcnt(0)\n\ts_barrier" ::: "memory")
#define RED_READ(bf, w) part[w] = *(LAS v4f*)((LAS char*)lds + (w) * LW + (bf) * 4096 + rd_off)
#define ADD4(a, b) do { _Pragma("unroll") for (int r_ = 0; r_ < 4; ++r_) { a[r_] += b[r_]; asm volatile("" : "+v"(a[r_])); } } while (0)
#define EPI_A(r) do { res[r] = part[0][r] * scale; \
        ex[r] = __builtin_amdgcn_exp2f(__builtin_amdgcn_fmed3f(res[r], -60.f, 60.f) * -1.44269504088896f); asm volatile("" : "+v"(res[r]), "+v"(ex[r])); } while (0)
#define EPI_B(r) do { res[r] *= __builtin_amdgcn_rcpf(1.f + ex[r]); asm volatile("" : "+v"(res[r])); } while (0)
#define STORE_RES(vo, so) __builtin_amdgcn_raw_buffer_store_b128(__builtin_bit_cast(u32x4, res), rs_out, vo, so, OUT_AUX)
#define MF(acc, t, ks) do { if (ks == 0) asm volatile("v_mfma_f32_16x16x128_f8f6f4 %0, %1, %2, %3 cbsz:1" : "=&v"(acc[t]) : "a"(wf[ks][t]), "v"(xv), "v"(cin)); \
        else asm volatile("v_mfma_f32_16x16x128_f8f6f4 %0, %1, %2, %0 cbsz:1" : "+v"(acc[t]) : "a"(wf[ks][t]), "v"(xv)); } while (0)
#define FILL(s, bf, accp, vo) do { \
        if (!(DBG_X & 1)) { \
        if (s == 1 || s == 3 || s == 5 || s == 7) RED_WRITE(accp, (bf) ^ 1, s >> 1); \
        if (s == RS0) RED_BAR(); \
        if (s == RS0 || s == RS0 + 1) { RED_READ((bf) ^ 1, 2 * (s - RS0)); RED_READ((bf) ^ 1, 2 * (s - RS0) + 1); } \
        if (s == RS0 + 4) ADD4(part[0], part[1]); \
        if (s == RS0 + 5) ADD4(part[2], part[3]); \
        if (s == RS0 + 6) ADD4(part[0], part[2]); } \
        if (!(DBG_X & 2)) { \
        if (s >= RS0 + 7 && s < RS0 + 11) EPI_A(s - RS0 - 7); \
        if (s >= RS0 + 11 && s < RS0 + 15) EPI_B(s - RS0 - 11); } \
        if (s == RS0 + 16 && !(DBG_X & 4)) STORE_RES(vo, oo_p); \
        } while (0)
#define XV(bf, ks) const u32x4 lo_ = xb[bf][2 * ks], hi_ = xb[bf][2 * ks + 1]; \
        const v8i xv = {(int)lo_[0], (int)lo_[1], (int)lo_[2], (int)lo_[3], (int)hi_[0], (int)hi_[1], (int)hi_[2], (int)hi_[3]}
#define OSTEP(j, bf, acc, accp) do { \
        so_d = OWN_SO((j) + 2); if ((bf) == 1) rs_own = mkrsrc(a_x, (j) < 3 ? XB : 0); \
        asm volatile("s_waitcnt lgkmcnt(0)" ::: "memory"); SB(); \
        _Pragma("unroll") for (int ks = 0; ks < 8; ++ks) { XV(bf, ks); \
            _Pragma("unroll") for (int t = 0; t < 4; ++t) { const int sl = 4 * ks + t; \
                MF(acc, t, ks); SB(); \
                if (sl == 14) waitv<15>(); \
                if (sl < 16) DMA(bf, sl); \
                if (sl >= 14 && sl < 30) XRD((bf) ^ 1, sl - 14); \
                FILL(sl, bf, accp, vo_st); \
                if (sl == 30) { vo_st = vo_nx; vo_nx = vo_out; asm volatile("" : "+v"(vo_st), "+v"(vo_nx)); } \
                if (sl == 1 && (bf) == 1) PLAN(1, 0); \
                if (t == 3 && (bf) == 1 && (j) == 3) { LD(1, 2 * ks); LD(1, 2 * ks + 1); } \
                SB(); } } \
        oo_p = oo_c; oo_c = oo_n; oo_n = oo_own + 2 * so_d; asm volatile("" : "+s"(oo_p), "+s"(oo_c), "+s"(oo_n)); \
        if (DBG_ALLSTEPS) TS(2 + (j)); \
        } while (0)
#define PK(op, d, a, b) asm volatile(op " %0, %1, %2" : "=v"(d) : "v"(a), "v"(b))
#define PKS(op, d, a, b) asm volatile(op " %0, %1, %2 op_sel_hi:[1,0]" : "=v"(d) : "v"(a), "s"(b))
#define LO(v) __builtin_shufflevector(v, v, 0, 1)
#define HI(v) __builtin_shufflevector(v, v, 2, 3)
#define RD2(bf, w) part[w] = *(LAS v4f*)((LAS char*)((w) < 2 ? rd_a : rd_b) + ((w) & 1) * LW + (bf) * 4096)
#define CLAMP(v, i) do { float f_; asm volatile("v_med3_f32 %0, %1, %2, %3" : "=v"(f_) : "v"(v[i]), "s"(-86.5617024533f), "v"(lim)); v[i] = f_; } while (0)
#define TR(op, v, i) do { float f_; asm volatile(op " %0, %1" : "=v"(f_) : "v"(v[i])); v[i] = f_; } while (0)
#define GSTEP(n, bf, acc, accp) do { \
        if (__builtin_expect(sp[bf] != 0, 0)) RETRY(bf, (n) - 2); \
        SB(); \
        _Pragma("unroll") for (int ks = 0; ks < 8; ++ks) { \
            XV(bf, ks); \
            _Pragma("unroll") for (int t = 0; t < 4; ++t) { const int sl = 4 * ks + t; \
                MF(acc, t, ks); SB(); \
                if (sl == 0) PLAN(bf, n); \
                if (sl == 1) RED_WRITE(accp, (bf) ^ 1, 0); \
                if (sl == 2) RED_WRITE(accp, (bf) ^ 1, 1); \
                if (sl == 2 && (bf) == 1 && (n) == zn) ZDONE(); \
                if (sl == 4) RED_WRITE(accp, (bf) ^ 1, 2); \
                if (sl == 5) RED_WRITE(accp, (bf) ^ 1, 3); \
                  \
                if (sl == 6) { ZW(18); ZM2((bf) ^ 1, 0); } \
                if (sl == 8) { RED_BAR(); RD2((bf) ^ 1, 0); RD2((bf) ^ 1, 1); RD2((bf) ^ 1, 2); RD2((bf) ^ 1, 3); } \
                if (sl == 9) { ZW(18); ZM2((bf) ^ 1, 2); } \
                if (sl == 11) { PK("v_pk_add_f32", qa, LO(part[0]), LO(part[1])); PK("v_pk_add_f32", qb, HI(part[0]), HI(part[1])); } \
                if (sl == 12) { PK("v_pk_add_f32", qc, LO(part[2]), LO(part[3])); PK("v_pk_add_f32", qd, HI(part[2]), HI(part[3])); \
                                PK("v_pk_add_f32", qa, qa, qc); PK("v_pk_add_f32", qb, qb, qd); } \
                if (sl == 13) { ZW(18); ZM2((bf) ^ 1, 4); } \
                if (sl == 14) { PKS("v_pk_mul_f32", qc, qa, nsl); PKS("v_pk_mul_f32", qd, qb, nsl); CLAMP(qc, 0); CLAMP(qc, 1); } \
                if (sl == 15) TR("v_exp_f32", qc, 0); \
                if (sl == 16) { TR("v_exp_f32", qc, 1); CLAMP(qd, 0); CLAMP(qd, 1); } \
                if (sl == 17) { TR("v_exp_f32", qd, 0); ZW(18); ZM((bf) ^ 1, 6); } \
                if (sl == 18) { TR("v_exp_f32", qd, 1); ZM((bf) ^ 1, 7); } \
                if (sl == 19) { PKS("v_pk_add_f32", qc, qc, one); TR("v_rcp_f32", qc, 0); } \
                if (sl == 20) { TR("v_rcp_f32", qc, 1); PKS("v_pk_add_f32", qd, qd, one); PKS("v_pk_mul_f32", qa, qa, scl); PKS("v_pk_mul_f32", qb, qb, scl); } \
                if (sl == 21) { TR("v_rcp_f32", qd, 0); ZW(18); ZM((bf) ^ 1, 8); } \
                if (sl == 22) { TR("v_rcp_f32", qd, 1); ZM((bf) ^ 1, 9); } \
                if (sl == 23) { PK("v_pk_mul_f32", qa, qa, qc); PK("v_pk_mul_f32", qb, qb, qd); res = __builtin_shufflevector(qa, qb, 0, 1, 2, 3); } \
                if (sl == 24) ZERO(); \
                if (sl == 25) { ZW(18); ZM2((bf) ^ 1, 10); } \
                if (sl == 26) STORE_RES(vo_out, oo[(bf) ^ 1]); \
                if (sl == 27) { oo[(bf) ^ 1] = 2 * so_q + oo_r1; asm volatile("" : "+s"(oo[(bf) ^ 1])); } \
                if (sl == 28) { ZW(18); ZM2((bf) ^ 1, 12); } \
                if (sl == 29) { ZW(16); ZM2((bf) ^ 1, 14); } \
                if (sl == 30) ZTEST(sp[(bf) ^ 1]); \
                if (t == 3) { LD(bf, 2 * ks); LD(bf, 2 * ks + 1); } \
                SB(); } } \
        if (DBG_ALLSTEPS) TS(5 + (n)); else if ((n) == 16) TS(21); \
        } while (0)

    POLL(pv);
    for (int j = 0; j < 4; j += 2) { OSTEP(j, 0, accA, accB); OSTEP(j + 1, 1, accB, accA); }
    if (!DBG_ALLSTEPS) TS(2);
    int oo[2] = {oo_c, oo_p};
    const int oo_r1 = oo_rem - 131072;
    v2f qa, qb, qc, qd;
    const unsigned long long scl = (unsigned)__builtin_amdgcn_readfirstlane(__builtin_bit_cast(int, scale)),
        nsl = (unsigned)__builtin_amdgcn_readfirstlane(__builtin_bit_cast(int, scale * -1.44269504088896f)), one = 0x3f800000u;
    float lim = 86.5617024533f;
    LAS char* rd_a = (LAS char*)lds + rd_off; LAS char* rd_b = rd_a + 2 * LW;
    asm volatile("" : "+v"(lim), "+v"(rd_a), "+v"(rd_b));
    int zn = wid == 0 ? 20 : -64;
    asm volatile("" : "+s"(zn));
    __builtin_amdgcn_s_waitcnt(0xc07f);
    for (int n = 0; n <= NBLK; n += 2) { if (n > 0) GSTEP(n, 1, accB, accA); GSTEP(n + 1, 0, accA, accB); }
    if (!DBG_ALLSTEPS) TS(39);
    asm volatile("s_nop 15" : "+v"(accA[0]), "+v"(accA[1]), "+v"(accA[2]), "+v"(accA[3]));
    RED_WRITE(accA, 0, 0); RED_WRITE(accA, 0, 1); RED_WRITE(accA, 0, 2); RED_WRITE(accA, 0, 3);
    RED_BAR(); RED_READ(0, 0); RED_READ(0, 1); RED_READ(0, 2); RED_READ(0, 3);
    ADD4(part[0], part[1]); ADD4(part[2], part[3]); ADD4(part[0], part[2]);
    EPI_A(0); EPI_A(1); EPI_A(2); EPI_A(3); EPI_B(0); EPI_B(1); EPI_B(2); EPI_B(3);
    STORE_RES(vo_out, oo[0]);
    SB();
    asm volatile("s_waitcnt vmcnt(0)" : "+v"(pv) :: "memory");
#pragma unroll
    for (int q = 0; q < 16; ++q) asm volatile("" :: "v"(xb[0][q]), "v"(xb[1][q]));
}

extern "C" void kernel_launch(int rank, int nranks, ncclComm_t comm, hipStream_t stream,
                              void* const* d_in, const int* in_sizes, int n_in,
                              void* d_out, int out_size, const int* in_dtypes, int out_dtype,
                              void* const* d_out_all, void* const* d_ws_all, size_t ws_bytes) {
    (void)comm; (void)in_dtypes; (void)out_dtype; (void)d_out_all;
    if (nranks != NR || rank < 0 || rank >= NR || n_in < 4 || ws_bytes < WS_NEED || in_sizes[0] != (int)XB || in_sizes[1] != KD * NC
        || out_size != NR * MS * NC) { fprintf(stderr, "unexpected problem shape\n"); return; }
    Params p;
    for (int i = 0; i < NR; ++i) p.dst[i] = (char*)d_ws_all[(rank + 1 + i) % NR];
    p.x = (const char*)d_in[0]; p.w = (const char*)d_in[1]; p.sx = (const float*)d_in[2]; p.sw = (const float*)d_in[3];
    p.out = (float*)d_out; p.rank = (unsigned)rank; p.pad = 0;
    ag_gemm_silu<<<GRID, 256, 0, stream>>>(p);
    HIP_CHECK(hipGetLastError());
}
```
